```python
import numpy as np

kernel_name = "gpu_gemm_a2a_m4096_k4096_n4096_e4m3e5m2_dqrelu"


ABSMAX_RTOL = 0.05

COMPUTE_DTYPE = "float32"
EPILOGUE = "dequant_relu"
IN_QUANT = None
OUT_QUANT = None
IN_FP8 = "e4m3e5m2"
IN_DTYPES = {"x": "e4m3", "w_mat": "e5m2",
             "scale_x": "float32", "scale_w": "float32"}
WORLD_SIZE = 8

def _snap_fp8(x, mant_bits, max_abs, min_normal):
    a = np.abs(x.astype(np.float64))
    _m, _e = np.frexp(a)
    q = np.where(a >= min_normal, np.ldexp(1.0, _e - 1 - mant_bits),
                 min_normal * 2.0 ** -mant_bits)
    snapped = np.minimum(np.round(a / q) * q, max_abs)
    return (np.sign(x) * snapped).astype(np.float32)

def setup_inputs():

    x = _snap_fp8(np.random.randn(4096, 4096).astype(np.float32), 3, 448.0, 2.0 ** -6)
    w_mat = _snap_fp8(np.random.randn(4096, 4096).astype(np.float32), 2, 57344.0, 2.0 ** -14)
    scale_x = (0.02 * (1.0 + np.random.rand(1))).astype(np.float32)
    scale_w = (0.02 * (1.0 + np.random.rand(1))).astype(np.float32)
    return {"x": x, "w_mat": w_mat, "scale_x": scale_x, "scale_w": scale_w}

def reference(x, w_mat, scale_x, scale_w):
    acc = (x @ w_mat).astype(np.float32)
    y = (acc * (scale_x * scale_w)).astype(np.float32)
    return np.maximum(y, 0.0).astype(np.float32)
```

```cpp
#include <hip/hip_runtime.h>
#include <rccl/rccl.h>
#include <cstdio>

#define HIP_CHECK(expr) do { hipError_t _e = (expr); if (_e != hipSuccess) { \
    fprintf(stderr, "HIP error %s at %s:%d\n", hipGetErrorString(_e), __FILE__, __LINE__); \
    return; } } while (0)

namespace {

typedef int v2i __attribute__((ext_vector_type(2)));
typedef int v4i __attribute__((ext_vector_type(4)));
typedef unsigned v4u __attribute__((ext_vector_type(4)));
typedef int v8i __attribute__((ext_vector_type(8)));
typedef float v4f __attribute__((ext_vector_type(4)));
#define LDS __attribute__((address_space(3)))

constexpr int NR = 8;
constexpr int MLOC = 512, NN = 4096, KK = 4096, NSH = 512;
constexpr int BM = 64, BN = 128, BK = 128, STAGES = 4;
constexpr int XB = BM * BK, WB = BK * BN, SB = XB + WB;
constexpr int NK = KK / BK;
constexpr int NWG = 256;
constexpr int LDS_BYTES = SB * STAGES;
constexpr unsigned long long DEADLINE_TICKS = 100000000ull;

constexpr unsigned WS_CTR = 0;
constexpr unsigned WS_TMO = 4096;
constexpr unsigned WS_FLAG = 8192;
constexpr unsigned WS_STAMP = 65536;

struct Params {
  const unsigned char* x;
  const unsigned char* w;
  const float* sx;
  const float* sw;
  float* out[NR];
  unsigned char* ws[NR];
  int rank;
};

template <int N_> __device__ __forceinline__ void wait_vm() { asm volatile("s_waitcnt vmcnt(%0)" :: "i"(N_) : "memory"); }

__device__ __forceinline__ void dma16(__amdgpu_buffer_rsrc_t rs, unsigned lds_dst, int voff, int soff) {
  asm volatile("s_mov_b32 m0, %0\n\ts_nop 0\n\tbuffer_load_dwordx4 %1, %2, %3 offen lds"
               :: "s"(lds_dst), "v"(voff), "s"(rs), "s"(soff) : "memory");
}
#define SCHED() __builtin_amdgcn_sched_barrier(0)

struct Frags { v8i xf[2]; v8i wf[4]; };

__global__ void __launch_bounds__(256, 2) gemm_a2a(Params p) {
  __shared__ __attribute__((aligned(1024))) unsigned char smem[LDS_BYTES];
  const int tid = threadIdx.x, lane = tid & 63;
  const int wv = __builtin_amdgcn_readfirstlane(tid >> 6);
  const int b = blockIdx.x;
  const int xcd = b & 7, idx = b >> 3;
  const int dest = (p.rank + xcd) & 7;
  const int tm = idx & 7, tn = idx >> 3;
  unsigned char* myws = p.ws[p.rank];
  const unsigned call = *(const unsigned*)(myws + WS_CTR + 4 * b) + 1;

  __amdgpu_buffer_rsrc_t rx = __builtin_amdgcn_make_buffer_rsrc((void*)(p.x + (size_t)tm * BM * KK), 0, BM * KK, 0x00020000);
  __amdgpu_buffer_rsrc_t rw = __builtin_amdgcn_make_buffer_rsrc((void*)(p.w + dest * NSH + tn * BN), 0, KK * NN - (dest * NSH + tn * BN), 0x00020000);

  int xo[2], wo[4];
#pragma unroll
  for (int i = 0; i < 2; ++i) { int row = (wv * 2 + i) * 8 + (lane >> 3); int c = (lane & 7) ^ ((row >> 1) & 7); xo[i] = row * KK + c * 16; }
#pragma unroll
  for (int i = 0; i < 4; ++i) { int k = (wv * 4 + i) * 8 + (lane >> 3); int f = ((k >> 1) & 3) | (((k >> 4) & 1) << 2); int c = (lane & 7) ^ f; wo[i] = k * NN + c * 16; }

  const unsigned smem_base = (unsigned)(size_t)(LDS unsigned char*)smem;
  auto issue = [&](int kt) {
    const unsigned sb = smem_base + (kt % STAGES) * SB;
#pragma unroll
    for (int i = 0; i < 2; ++i) dma16(rx, sb + (wv * 2 + i) * 1024, xo[i], kt * BK);
#pragma unroll
    for (int i = 0; i < 4; ++i) dma16(rw, sb + XB + (wv * 4 + i) * 1024, wo[i], kt * BK * NN);
  };

  const int wm = wv & 1, wn = wv >> 1;
  const int r = lane & 15, g = lane >> 4;
  const int xa0 = (wm * 32 + r) * 128 + ((g ^ (r >> 1)) << 4);
  const int xa1 = xa0 ^ 64;
  int wa[4];
  {
    const int q = r >> 1, pp = r & 1, fl = ((q >> 1) & 3) | ((g & 1) << 2);
#pragma unroll
    for (int ni = 0; ni < 4; ++ni) wa[ni] = XB + (16 * g + q) * 128 + (((wn * 4 + ni) ^ fl) << 4) + pp * 8;
  }
  auto rd = [&](Frags& f, int kt) {
    const int sb = (kt % STAGES) * SB;
#pragma unroll
    for (int ni = 0; ni < 4; ++ni) {
#pragma unroll
      for (int j = 0; j < 4; ++j) {
        v2i t = __builtin_amdgcn_ds_read_tr8_b64_v2i32((LDS v2i*)(smem + sb + wa[ni] + (j & 1) * 1024 + (j >> 1) * 8192));
        f.wf[ni][2 * j] = t[0]; f.wf[ni][2 * j + 1] = t[1];
      }
    }
#pragma unroll
    for (int mi = 0; mi < 2; ++mi) {
      v4i a = *(LDS v4i*)(smem + sb + xa0 + mi * 2048);
      v4i c = *(LDS v4i*)(smem + sb + xa1 + mi * 2048);
      f.xf[mi] = __builtin_shufflevector(a, c, 0, 1, 2, 3, 4, 5, 6, 7);
    }
  };
  v4f acc[2][4];
#pragma unroll
  for (int mi = 0; mi < 2; ++mi)
#pragma unroll
    for (int ni = 0; ni < 4; ++ni) acc[mi][ni] = (v4f){0.f, 0.f, 0.f, 0.f};
  auto mma = [&](const Frags& f) {
#pragma unroll
    for (int ni = 0; ni < 4; ++ni)
#pragma unroll
      for (int mi = 0; mi < 2; ++mi)
        acc[mi][ni] = __builtin_amdgcn_mfma_scale_f32_16x16x128_f8f6f4(f.wf[ni], f.xf[mi], acc[mi][ni],  1,  0, 0, 0, 0, 0);
  };

#pragma unroll
  for (int s = 0; s < STAGES - 1; ++s) issue(s);
  Frags f0, f1;
  wait_vm<6 * (STAGES - 2)>();
  __builtin_amdgcn_s_barrier();
  rd(f0, 0);
  int kt = 0;
  for (; kt + STAGES < NK; kt += 2) {
    wait_vm<6 * (STAGES - 3)>();
    __builtin_amdgcn_s_barrier();
    issue(kt + STAGES - 1);
    rd(f1, kt + 1);
    SCHED();
    mma(f0);
    SCHED();
    wait_vm<6 * (STAGES - 3)>();
    __builtin_amdgcn_s_barrier();
    issue(kt + STAGES);
    rd(f0, kt + 2);
    SCHED();
    mma(f1);
    SCHED();
  }
  static_assert(STAGES == 4, "tail written for 4 stages");
  wait_vm<6>(); __builtin_amdgcn_s_barrier(); issue(NK - 1); rd(f1, NK - 3); SCHED(); mma(f0); SCHED();
  wait_vm<6>(); __builtin_amdgcn_s_barrier(); rd(f0, NK - 2); SCHED(); mma(f1); SCHED();
  wait_vm<0>(); __builtin_amdgcn_s_barrier(); rd(f1, NK - 1); SCHED(); mma(f0); SCHED();
  mma(f1);
  const float scale = p.sx[0] * p.sw[0];
  __amdgpu_buffer_rsrc_t ro = __builtin_amdgcn_make_buffer_rsrc((void*)(p.out[dest] + (size_t)p.rank * MLOC * NSH), 0, MLOC * NSH * 4, 0x00020000);
  const int obase = ((tm * BM + wm * 32 + r) * NSH + tn * BN + wn * 64 + g * 4) * 4;
#pragma unroll
  for (int mi = 0; mi < 2; ++mi)
#pragma unroll
    for (int ni = 0; ni < 4; ++ni) {
      v4f y = acc[mi][ni] * scale;
#pragma unroll
      for (int j = 0; j < 4; ++j) y[j] = (y[j] <= 0.f) ? 0.f : y[j];
      __builtin_amdgcn_raw_buffer_store_b128(__builtin_bit_cast(v4u, y), ro, obase + mi * 16 * NSH * 4 + ni * 64, 0, 17);
    }
  wait_vm<0>();
  __syncthreads();
  if (wv == 0) {
    if (tid == 0) {
      if (dest != p.rank)
        __hip_atomic_store((unsigned*)(p.ws[dest] + WS_FLAG) + p.rank * 32 + idx, call, __ATOMIC_RELAXED, __HIP_MEMORY_SCOPE_SYSTEM);
      *((unsigned*)(myws + WS_CTR) + b) = call;
    }
    if (xcd == 0 && lane < NR && lane != p.rank) {
      unsigned* f = (unsigned*)(myws + WS_FLAG) + lane * 32 + idx;
      unsigned* tmo = (unsigned*)(myws + WS_TMO);
      unsigned long long ts = __builtin_amdgcn_s_memrealtime();
      while (__hip_atomic_load(f, __ATOMIC_RELAXED, __HIP_MEMORY_SCOPE_SYSTEM) < call) {
        if (__hip_atomic_load(tmo, __ATOMIC_RELAXED, __HIP_MEMORY_SCOPE_AGENT) == call ||
            __builtin_amdgcn_s_memrealtime() - ts > DEADLINE_TICKS) {
          __hip_atomic_store(tmo, call, __ATOMIC_RELAXED, __HIP_MEMORY_SCOPE_AGENT);
          p.out[p.rank][(size_t)p.rank * MLOC * NSH + (tm * BM) * NSH + tn * BN] = __builtin_nanf("");
          break;
        }
        __builtin_amdgcn_s_sleep(1);
      }
    }
  }
}

}

extern "C" void kernel_launch(int rank, int nranks, ncclComm_t comm, hipStream_t stream,
                              void* const* d_in, const int* in_sizes, int n_in,
                              void* d_out, int out_size, const int* in_dtypes, int out_dtype,
                              void* const* d_out_all, void* const* d_ws_all, size_t ws_bytes) {
  if (nranks != NR) return;
  Params p;
  p.x = (const unsigned char*)d_in[0];
  p.w = (const unsigned char*)d_in[1];
  p.sx = (const float*)d_in[2];
  p.sw = (const float*)d_in[3];
  for (int i = 0; i < NR; ++i) { p.out[i] = (float*)d_out_all[i]; p.ws[i] = (unsigned char*)d_ws_all[i]; }
  p.rank = rank;
  gemm_a2a<<<NWG, 256, 0, stream>>>(p);
  HIP_CHECK(hipGetLastError());
}
```

```cpp
#include <hip/hip_runtime.h>
#include <rccl/rccl.h>
#include <cstdio>

#define HIP_CHECK(expr) do { hipError_t _e = (expr); if (_e != hipSuccess) { \
    fprintf(stderr, "HIP error %s at %s:%d\n", hipGetErrorString(_e), __FILE__, __LINE__); \
    return; } } while (0)

namespace {

typedef int v2i __attribute__((ext_vector_type(2)));
typedef int v4i __attribute__((ext_vector_type(4)));
typedef unsigned v4u __attribute__((ext_vector_type(4)));
typedef int v8i __attribute__((ext_vector_type(8)));
typedef float v4f __attribute__((ext_vector_type(4)));
#define LDS __attribute__((address_space(3)))

constexpr int NR = 8;
constexpr int MLOC = 512, NN = 4096, KK = 4096, NSH = 512;
constexpr int BM = 64, BN = 128, BK = 128, STAGES = 5;
constexpr int XB = BM * BK, WB = BK * BN, SB = XB + WB;
constexpr int NK = KK / BK;
constexpr int NWG = 256;
constexpr int LDS_BYTES = SB * STAGES;
constexpr unsigned long long DEADLINE_TICKS = 100000000ull;

constexpr unsigned WS_CTR = 0;
constexpr unsigned WS_TMO = 4096;
constexpr unsigned WS_FLAG = 8192;
constexpr unsigned WS_STAMP = 65536;

struct Params {
  const unsigned char* x;
  const unsigned char* w;
  const float* sx;
  const float* sw;
  float* out[NR];
  unsigned char* ws[NR];
  int rank;
};

template <int N_> __device__ __forceinline__ void wait_vm() { asm volatile("s_waitcnt vmcnt(%0)" :: "i"(N_) : "memory"); }

__device__ __forceinline__ void dma16(__amdgpu_buffer_rsrc_t rs, unsigned lds_dst, int voff, int soff) {
  asm volatile("s_mov_b32 m0, %0\n\ts_nop 0\n\tbuffer_load_dwordx4 %1, %2, %3 offen lds"
               :: "s"(lds_dst), "v"(voff), "s"(rs), "s"(soff) : "memory");
}
#define SCHED() __builtin_amdgcn_sched_barrier(0)
#define BAR() do { SCHED(); asm volatile("s_barrier" ::: "memory"); SCHED(); } while (0)

struct Frags { v8i xf[2]; v8i wf[4]; };

__global__ void __launch_bounds__(512, 2) gemm_a2a(Params p) {
  __shared__ __attribute__((aligned(1024))) unsigned char smem[LDS_BYTES];
  const int tid = threadIdx.x, lane = tid & 63;
  const int wv8 = __builtin_amdgcn_readfirstlane(tid >> 6);
  const int wv = wv8 & 3;
  const int b = blockIdx.x;
  const int xcd = b & 7, idx = b >> 3;
  const int dest = (p.rank + xcd) & 7;
  const int tm = idx & 7, tn = idx >> 3;
  unsigned char* myws = p.ws[p.rank];

  if (wv8 >= 4) {
    __amdgpu_buffer_rsrc_t rx = __builtin_amdgcn_make_buffer_rsrc((void*)(p.x + (size_t)tm * BM * KK), 0, BM * KK, 0x00020000);
    __amdgpu_buffer_rsrc_t rw = __builtin_amdgcn_make_buffer_rsrc((void*)(p.w + dest * NSH + tn * BN), 0, KK * NN - (dest * NSH + tn * BN), 0x00020000);
    int xo[2], wo[4];
#pragma unroll
    for (int i = 0; i < 2; ++i) { int row = (wv * 2 + i) * 8 + (lane >> 3); int c = (lane & 7) ^ ((row >> 1) & 7); xo[i] = row * KK + c * 16; }
#pragma unroll
    for (int i = 0; i < 4; ++i) { int k = (wv * 4 + i) * 8 + (lane >> 3); int f = ((k >> 1) & 3) | (((k >> 4) & 1) << 2); int c = (lane & 7) ^ f; wo[i] = k * NN + c * 16; }
    const unsigned smem_base = (unsigned)(size_t)(LDS unsigned char*)smem;
    const int k0 = ((tm + xcd) & 7) * (NK / 8);
    auto issue = [&](int ks) {
      const unsigned sb = smem_base + (ks % STAGES) * SB;
      const int kt = (ks + k0) & (NK - 1);
#pragma unroll
      for (int i = 0; i < 4; ++i) dma16(rw, sb + XB + (wv * 4 + i) * 1024, wo[i], kt * BK * NN);
#pragma unroll
      for (int i = 0; i < 2; ++i) dma16(rx, sb + (wv * 2 + i) * 1024, xo[i], kt * BK);
    };
    for (int s = 0; s < STAGES - 1; ++s) issue(s);
    wait_vm<6 * (STAGES - 2)>();
    __builtin_amdgcn_s_barrier();
    for (int kt = 0; kt + STAGES - 1 < NK; ++kt) {
      wait_vm<6 * (STAGES - 3)>();
      __builtin_amdgcn_s_barrier();
      issue(kt + STAGES - 1);
    }
    static_assert(STAGES == 5, "tail");
    wait_vm<12>(); __builtin_amdgcn_s_barrier();
    wait_vm<6>(); __builtin_amdgcn_s_barrier();
    wait_vm<0>(); __builtin_amdgcn_s_barrier();
    return;
  }
  const unsigned call = *(const unsigned*)(myws + WS_CTR + 4 * b) + 1;

  const int wm = wv & 1, wn = wv >> 1;
  const int r = lane & 15, g = lane >> 4;
  const int xa0 = (wm * 32 + r) * 128 + ((g ^ (r >> 1)) << 4);
  const int xa1 = xa0 ^ 64;
  int wa[4];
  {
    const int q = r >> 1, pp = r & 1, fl = ((q >> 1) & 3) | ((g & 1) << 2);
#pragma unroll
    for (int ni = 0; ni < 4; ++ni) wa[ni] = XB + (16 * g + q) * 128 + (((wn * 4 + ni) ^ fl) << 4) + pp * 8;
  }
  auto rd = [&](Frags& f, int sb) {
#pragma unroll
    for (int mi = 0; mi < 2; ++mi) {
      v4i a = *(LDS v4i*)(smem + sb + xa0 + mi * 2048);
      v4i c = *(LDS v4i*)(smem + sb + xa1 + mi * 2048);
      f.xf[mi] = __builtin_shufflevector(a, c, 0, 1, 2, 3, 4, 5, 6, 7);
    }
#pragma unroll
    for (int ni = 0; ni < 4; ++ni) {
#pragma unroll
      for (int j = 0; j < 4; ++j) {
        v2i t = __builtin_amdgcn_ds_read_tr8_b64_v2i32((LDS v2i*)(smem + sb + wa[ni] + (j & 1) * 1024 + (j >> 1) * 8192));
        f.wf[ni][2 * j] = t[0]; f.wf[ni][2 * j + 1] = t[1];
      }
    }
  };
  v4f acc[2][4];
#pragma unroll
  for (int mi = 0; mi < 2; ++mi)
#pragma unroll
    for (int ni = 0; ni < 4; ++ni) acc[mi][ni] = (v4f){0.f, 0.f, 0.f, 0.f};
  auto mma = [&](const Frags& f) {
#pragma unroll
    for (int ni = 0; ni < 4; ++ni)
#pragma unroll
      for (int mi = 0; mi < 2; ++mi)
        acc[mi][ni] = __builtin_amdgcn_mfma_scale_f32_16x16x128_f8f6f4(f.wf[ni], f.xf[mi], acc[mi][ni],  1,  0, 0, 0, 0, 0);
  };
  auto interleave = [&]() {
#pragma unroll
    for (int i = 0; i < 8; ++i) {
      __builtin_amdgcn_sched_group_barrier(0x008, 1, 0);
      __builtin_amdgcn_sched_group_barrier(0x002, 1, 0);
      __builtin_amdgcn_sched_group_barrier(0x100, 3, 0);
    }
  };
  Frags f0, f1;
  BAR();
  rd(f0, 0);
  int sb = SB;
  for (int kt = 0; kt + 2 < NK; kt += 2) {
    BAR();
    rd(f1, sb); mma(f0); interleave();
    sb += SB; if (sb == SB * STAGES) sb = 0;
    BAR();
    rd(f0, sb); mma(f1); interleave();
    sb += SB; if (sb == SB * STAGES) sb = 0;
  }
  BAR();
  rd(f1, sb); mma(f0); interleave();
  SCHED();
  mma(f1);
  const float scale = p.sx[0] * p.sw[0];
  __amdgpu_buffer_rsrc_t ro = __builtin_amdgcn_make_buffer_rsrc((void*)(p.out[dest] + (size_t)p.rank * MLOC * NSH), 0, MLOC * NSH * 4, 0x00020000);
  const int obase = ((tm * BM + wm * 32 + r) * NSH + tn * BN + wn * 64 + g * 4) * 4;
#pragma unroll
  for (int mi = 0; mi < 2; ++mi)
#pragma unroll
    for (int ni = 0; ni < 4; ++ni) {
      v4f y = acc[mi][ni] * scale;
#pragma unroll
      for (int j = 0; j < 4; ++j) y[j] = (y[j] <= 0.f) ? 0.f : y[j];
      __builtin_amdgcn_raw_buffer_store_b128(__builtin_bit_cast(v4u, y), ro, obase + mi * 16 * NSH * 4 + ni * 64, 0, 17);
    }
  wait_vm<0>();
  BAR();
  if (wv == 0) {
    if (tid == 0) {
      if (dest != p.rank)
        __hip_atomic_store((unsigned*)(p.ws[dest] + WS_FLAG) + p.rank * 32 + idx, call, __ATOMIC_RELAXED, __HIP_MEMORY_SCOPE_SYSTEM);
      *((unsigned*)(myws + WS_CTR) + b) = call;
    }
    if (xcd == 0 && lane < NR && lane != p.rank) {
      unsigned* f = (unsigned*)(myws + WS_FLAG) + lane * 32 + idx;
      unsigned* tmo = (unsigned*)(myws + WS_TMO);
      unsigned long long ts = __builtin_amdgcn_s_memrealtime();
      while (__hip_atomic_load(f, __ATOMIC_RELAXED, __HIP_MEMORY_SCOPE_SYSTEM) < call) {
        if (__hip_atomic_load(tmo, __ATOMIC_RELAXED, __HIP_MEMORY_SCOPE_AGENT) == call ||
            __builtin_amdgcn_s_memrealtime() - ts > DEADLINE_TICKS) {
          __hip_atomic_store(tmo, call, __ATOMIC_RELAXED, __HIP_MEMORY_SCOPE_AGENT);
          p.out[p.rank][(size_t)p.rank * MLOC * NSH + (tm * BM) * NSH + tn * BN] = __builtin_nanf("");
          break;
        }
        __builtin_amdgcn_s_sleep(1);
      }
    }
  }
}

}

extern "C" void kernel_launch(int rank, int nranks, ncclComm_t comm, hipStream_t stream,
                              void* const* d_in, const int* in_sizes, int n_in,
                              void* d_out, int out_size, const int* in_dtypes, int out_dtype,
                              void* const* d_out_all, void* const* d_ws_all, size_t ws_bytes) {
  if (nranks != NR) return;
  Params p;
  p.x = (const unsigned char*)d_in[0];
  p.w = (const unsigned char*)d_in[1];
  p.sx = (const float*)d_in[2];
  p.sw = (const float*)d_in[3];
  for (int i = 0; i < NR; ++i) { p.out[i] = (float*)d_out_all[i]; p.ws[i] = (unsigned char*)d_ws_all[i]; }
  p.rank = rank;
  gemm_a2a<<<NWG, 512, 0, stream>>>(p);
  HIP_CHECK(hipGetLastError());
}
```

```cpp
#include <hip/hip_runtime.h>
#include <rccl/rccl.h>
#include <cstdio>
#include <cstdlib>

#define HIP_CHECK(expr) do { hipError_t _e = (expr); if (_e != hipSuccess) { \
    fprintf(stderr, "HIP error %s at %s:%d\n", hipGetErrorString(_e), __FILE__, __LINE__); \
    return; } } while (0)

namespace {

typedef int v2i __attribute__((ext_vector_type(2)));
typedef int v4i __attribute__((ext_vector_type(4)));
typedef unsigned v4u __attribute__((ext_vector_type(4)));
typedef int v8i __attribute__((ext_vector_type(8)));
typedef float v4f __attribute__((ext_vector_type(4)));
typedef unsigned long long u64;
#define LDS __attribute__((address_space(3)))

constexpr int NR = 8;
constexpr int MLOC = 512, NN = 4096, KK = 4096, NSH = 512;
constexpr int STAGES = 4;
constexpr int XB = 8192, WOFF = 2 * XB, SB = 32768;
constexpr int NWG = 256;
constexpr u64 DEADLINE_TICKS = 100000000ull;

constexpr int HDR_DW = 34;
constexpr int SLOT = 4352;
constexpr int UNITS = 256;
constexpr int PAIR_BYTES = UNITS * SLOT;
constexpr int MSG_OFF = SB * STAGES;
constexpr int MISC_OFF = MSG_OFF + 4 * SLOT;
constexpr int LDS_BYTES = MISC_OFF + 512;
constexpr unsigned WS_CTR = 0;
constexpr unsigned WS_TMO = 4096;
constexpr unsigned WS_STAMP = 65536;
constexpr unsigned WS_STAGE = 1u << 20;
constexpr unsigned WS_NEED = WS_STAGE + 2u * NR * PAIR_BYTES;
constexpr int AUX_SYS = 17;
constexpr int AUX_WT = 16;

struct Params {
  const unsigned char* x;
  const unsigned char* w;
  unsigned char* myws;
  float* out;
  const float* sx;
  const float* sw;
  int rank, thr, pfstart, pad;
  unsigned char* ws[NR];
  unsigned grp[8];
  unsigned tab[NWG][2];
};

template <int N_> __device__ __forceinline__ void wait_vm() { asm volatile("s_waitcnt vmcnt(%0)" :: "i"(N_) : "memory"); }
__device__ __forceinline__ void dma16(__amdgpu_buffer_rsrc_t rs, unsigned lds_dst, int voff, int soff) {
  asm volatile("s_mov_b32 m0, %0\n\ts_nop 0\n\tbuffer_load_dwordx4 %1, %2, %3 offen lds"
               :: "s"(lds_dst), "v"(voff), "s"(rs), "s"(soff) : "memory");
}
__device__ __forceinline__ void touch4(__amdgpu_buffer_rsrc_t rs, unsigned lds_dst, int voff, int soff) {
  asm volatile("s_mov_b32 m0, %0\n\ts_nop 0\n\tbuffer_load_dword %1, %2, %3 offen lds"
               :: "s"(lds_dst), "v"(voff), "s"(rs), "s"(soff) : "memory");
}
#define SCHED() __builtin_amdgcn_sched_barrier(0)
#define BAR() do { SCHED(); asm volatile("s_barrier" ::: "memory"); SCHED(); } while (0)
#define LBAR() do { SCHED(); __builtin_amdgcn_s_waitcnt(0xC07F); asm volatile("s_barrier" ::: "memory"); SCHED(); } while (0)
__device__ __forceinline__ unsigned mbcnt(u64 m, unsigned base) {
  return __builtin_amdgcn_mbcnt_hi((unsigned)(m >> 32), __builtin_amdgcn_mbcnt_lo((unsigned)m, base));
}
extern "C" __device__ unsigned wrlane(unsigned val, unsigned lane, unsigned old) __asm("llvm.amdgcn.writelane.i32");

struct Frags { v8i xf[2]; v8i wf[4]; };

#define STAMP(i)

__device__ __forceinline__ void warm_l2(const Params& p, unsigned h0, unsigned h1, int wv8, int lane, unsigned junk) {
  const int ns = (h0 >> 25) & 7;
  if (ns == 0) return;
  __amdgpu_buffer_rsrc_t rx = __builtin_amdgcn_make_buffer_rsrc((void*)p.x, 0, MLOC * KK, 0x00020000);
  __amdgpu_buffer_rsrc_t rw = __builtin_amdgcn_make_buffer_rsrc((void*)p.w, 0, KK * NN, 0x00020000);
  const int nx = (h1 >> 12) & 15;
  const int per = 2 * ns + nx;
  const int nw = ((h1 >> 6) & 63) * 8;
  const int voff = lane * 4096;
  int j = (h1 & 63) * 8 + wv8, kc = p.pfstart;
  for (;;) {
    while (j >= per) { j -= per; ++kc; }
    if (kc >= 32) break;
    if (j < 2 * ns) touch4(rw, junk, voff, (kc * 128 + (j & 1) * 64) * NN + ((h0 >> (5 * (j >> 1))) & 31) * 128);
    else touch4(rx, junk, voff, (j - 2 * ns) * 64 * KK + kc * 128);
    j += nw;
  }
}

__global__ void __launch_bounds__(512, 2) gemm_a2a(Params p) {
  __shared__ __attribute__((aligned(1024))) unsigned char smem[LDS_BYTES];
  const int tid = threadIdx.x, lane = tid & 63;
  const int wv8 = __builtin_amdgcn_readfirstlane(tid >> 6);
  const int wv = wv8 & 3;
  const int b = blockIdx.x;
  const unsigned smem_base = (unsigned)(size_t)(LDS unsigned char*)smem;
  const unsigned td0 = p.tab[b][0], td1 = p.tab[b][1];
  unsigned call;

  if (wv8 >= 4) {
    __amdgpu_buffer_rsrc_t rx = __builtin_amdgcn_make_buffer_rsrc((void*)p.x, 0, MLOC * KK, 0x00020000);
    __amdgpu_buffer_rsrc_t rw = __builtin_amdgcn_make_buffer_rsrc((void*)p.w, 0, KK * NN, 0x00020000);
    const int l3 = lane >> 3, l7 = lane & 7;
    const int xo0 = (wv * 16 + l3) * KK + ((l7 ^ (l3 >> 1)) << 4);
    const int xo1 = (xo0 + 8 * KK) ^ 64;
    const int c0 = l7 ^ (l3 >> 1), c1 = c0 ^ 4;
    u64 tstart = 0;
    if (td0 >> 16) {
      tstart = __builtin_amdgcn_s_memrealtime();
      warm_l2(p, p.grp[b & 7], td1 >> 11, wv8, lane, smem_base + MISC_OFF + 256);
    }
    unsigned td = td0;
#pragma nounroll
    for (int ti = 0; ti < 2; ++ti) {
      const bool E = td & 512;
      const int krow = wv * 32 + l3;
      const int wo0 = E ? (krow + (c0 >> 2) * 128) * NN + (c0 & 3) * 16 : krow * NN + c0 * 16;
      const int wo2 = (E ? (krow + (c1 >> 2) * 128) * NN + (c1 & 3) * 16 : krow * NN + c1 * 16) + 16 * NN;
      const int wo1 = wo0 + 8 * NN, wo3 = wo2 + 8 * NN;
      const int xbase = ((td >> 6) & 7) * 64 * KK, wbase = (td & 63) * 64;
      const int ksh = E ? 8 : 7;
      int kpos = ((td >> 11) & 7) << 9;
      unsigned dst = smem_base + wv * 2048;
      auto issue = [&]() {
        const int so = wbase + kpos * NN, sx = xbase + kpos;
        const unsigned dw = dst + WOFF + wv * 2048;
        dma16(rw, dw, wo0, so); dma16(rw, dw + 1024, wo1, so); dma16(rw, dw + 2048, wo2, so); dma16(rw, dw + 3072, wo3, so);
        dma16(rx, dst, xo0, sx); dma16(rx, dst + 1024, xo1, sx);
        if (E) { dma16(rx, dst + XB, xo0, sx + 128); dma16(rx, dst + XB + 1024, xo1, sx + 128); }
        kpos = (kpos + (1 << ksh)) & (KK - 1);
        dst = smem_base + ((dst - smem_base + SB) & (STAGES * SB - 1));
      };
      if (ti == 0 && (td >> 16)) {
        while ((unsigned)(__builtin_amdgcn_s_memrealtime() - tstart) < (td >> 16)) __builtin_amdgcn_s_sleep(2);
        wait_vm<0>();
      }
      if (ti == 0) STAMP(1);
#pragma nounroll
      for (int s = 0; s < STAGES; ++s) issue();
      if (E) wait_vm<24>(); else wait_vm<18>();
      if (ti == 0) STAMP(2);
      BAR();
      const int nmain = (KK >> ksh) - STAGES;
#pragma nounroll
      for (int kt = 0; kt < nmain; ++kt) {
        if (E) wait_vm<16>(); else wait_vm<12>();
        BAR();
        issue();
      }
      static_assert(STAGES == 4, "tail");
      if (E) wait_vm<16>(); else wait_vm<12>();
      BAR();
      if (E) wait_vm<8>(); else wait_vm<6>();
      BAR();
      wait_vm<0>();
      if (ti == 0) STAMP(3);
      BAR();
      BAR();
      td = td1;
      if (!(td & 1024)) break;
    }
    call = *(volatile LDS unsigned*)(smem + MISC_OFF);
  } else {
    if (td0 >> 16) warm_l2(p, p.grp[b & 7], td1 >> 11, wv8, lane, smem_base + MISC_OFF + 256);
    call = __builtin_amdgcn_readfirstlane(__hip_atomic_load((const unsigned*)(p.myws + WS_CTR) + b, __ATOMIC_RELAXED, __HIP_MEMORY_SCOPE_AGENT)) + 1;
    const float scale = p.sx[0] * p.sw[0];
    if (tid == 0) *(volatile LDS unsigned*)(smem + MISC_OFF) = call;
    const int r = lane & 15, g = lane >> 4;
    const int wm = wv & 1, wn = wv >> 1;
    int wa[4];
    {
      const int q = r >> 1, pp = r & 1, fl = ((q >> 1) & 3) | ((g & 1) << 2);
#pragma unroll
      for (int ni = 0; ni < 4; ++ni) wa[ni] = WOFF + (16 * g + q) * 128 + (((wn * 4 + ni) ^ fl) << 4) + pp * 8;
    }
    unsigned td = td0;
#pragma nounroll
    for (int ti = 0; ti < 2; ++ti) {
      const bool E = td & 512;
      const int dest = (td >> 3) & 7, tm = (td >> 6) & 7, cb = td & 7;
      const int xa0 = (wm * 32 + r) * 128 + ((g ^ (r >> 1)) << 4) + ((E && wn) ? XB : 0);
      const int xa1 = xa0 ^ 64;
      auto rd = [&](Frags& f, int sb) {
#pragma unroll
        for (int mi = 0; mi < 2; ++mi) {
          v4i a = *(LDS v4i*)(smem + sb + xa0 + mi * 2048);
          v4i c = *(LDS v4i*)(smem + sb + xa1 + mi * 2048);
          f.xf[mi] = __builtin_shufflevector(a, c, 0, 1, 2, 3, 4, 5, 6, 7);
        }
#pragma unroll
        for (int ni = 0; ni < 4; ++ni) {
#pragma unroll
          for (int j = 0; j < 4; ++j) {
            v2i t = __builtin_amdgcn_ds_read_tr8_b64_v2i32((LDS v2i*)(smem + sb + wa[ni] + (j & 1) * 1024 + (j >> 1) * 8192));
            f.wf[ni][2 * j] = t[0]; f.wf[ni][2 * j + 1] = t[1];
          }
        }
      };
      v4f acc[2][4];
#pragma unroll
      for (int mi = 0; mi < 2; ++mi)
#pragma unroll
        for (int ni = 0; ni < 4; ++ni) acc[mi][ni] = (v4f){0.f, 0.f, 0.f, 0.f};
      auto mma = [&](const Frags& f) {
#pragma unroll
        for (int ni = 0; ni < 4; ++ni)
#pragma unroll
          for (int mi = 0; mi < 2; ++mi)
            acc[mi][ni] = __builtin_amdgcn_mfma_scale_f32_16x16x128_f8f6f4(f.wf[ni], f.xf[mi], acc[mi][ni],  1,  0, 0, 0, 0, 0);
      };
      auto interleave = [&]() {
#pragma unroll
        for (int i = 0; i < 8; ++i) {
          __builtin_amdgcn_sched_group_barrier(0x008, 1, 0);
          __builtin_amdgcn_sched_group_barrier(0x002, 2, 0);
          __builtin_amdgcn_sched_group_barrier(0x100, 4, 0);
        }
      };
      const int ns = E ? 16 : 32;
      Frags f0, f1;
      LBAR();
      rd(f0, 0);
      int sb = 0;
      for (int kt = 0; kt + 2 < ns; kt += 2) {
        LBAR();
        rd(f1, sb + SB); mma(f0); interleave();
        sb ^= 2 * SB;
        LBAR();
        rd(f0, sb); mma(f1); interleave();
      }
      LBAR();
      rd(f1, sb + SB); mma(f0); interleave();
      SCHED();
      mma(f1);
      SCHED();
      int nu = 2;
      if (E) {
        LDS v4f* xs = (LDS v4f*)(smem + MSG_OFF + wv * SLOT) + lane;
        LDS v4f* xr = (LDS v4f*)(smem + MSG_OFF + (wv ^ 2) * SLOT) + lane;
#pragma unroll
        for (int mi = 0; mi < 2; ++mi)
#pragma unroll
          for (int nj = 0; nj < 2; ++nj) {
            xs[(mi * 2 + nj) * 64] = wn ? acc[mi][nj] : acc[mi][2 + nj];
            if (wn) acc[mi][nj] = acc[mi][2 + nj];
          }
        LBAR();
#pragma unroll
        for (int mi = 0; mi < 2; ++mi)
#pragma unroll
          for (int nj = 0; nj < 2; ++nj) acc[mi][nj] += xr[(mi * 2 + nj) * 64];
        nu = 1;
      } else {
        BAR();
      }
      if (ti == 0) STAMP(1); else STAMP(3);
      const int ucol0 = 2 * cb + (E ? wn : 2 * wn), urow = tm * 2 + wm;
      if (dest == p.rank) {
        __amdgpu_buffer_rsrc_t ro = __builtin_amdgcn_make_buffer_rsrc((void*)p.out, 0, NR * MLOC * NSH * 4, 0x00020000);
        const int obase = ((p.rank * MLOC + urow * 32 + r) * NSH + ucol0 * 32 + g * 4) * 4;
#pragma unroll
        for (int ni = 0; ni < 4; ++ni) {
          if (ni < 2 * nu) {
#pragma unroll
            for (int mi = 0; mi < 2; ++mi) {
              v4f y = acc[mi][ni] * scale;
#pragma unroll
              for (int j = 0; j < 4; ++j) y[j] = (y[j] <= 0.f) ? 0.f : y[j];
              __builtin_amdgcn_raw_buffer_store_b128(__builtin_bit_cast(v4u, y), ro, obase + mi * 16 * NSH * 4 + ni * 64, 0, AUX_WT);
            }
          }
        }
      } else {
        __amdgpu_buffer_rsrc_t rd_ = __builtin_amdgcn_make_buffer_rsrc(
            (void*)(p.ws[dest] + WS_STAGE + ((call & 1) * NR + p.rank) * PAIR_BYTES + (urow * 16 + ucol0) * SLOT), 0, 2 * SLOT, 0x00020000);
        LDS unsigned* msg = (LDS unsigned*)(smem + MSG_OFF + (E ? (wv ^ 2) : wv) * SLOT);
        const unsigned msg_base = (unsigned)(size_t)msg;
        const int thr = (td1 & 1024) && ti == 0 ? 0 : p.thr;
#pragma nounroll
        for (int h = 0; h < nu; ++h) {
          unsigned wp = msg_base + HDR_DW * 4;
          unsigned hdr = 0;
#pragma unroll
          for (int mi = 0; mi < 2; ++mi)
#pragma unroll
            for (int nj = 0; nj < 2; ++nj) {
              const v4f y = acc[mi][nj] * scale;
              acc[mi][nj] = acc[mi][2 + nj];
              const float y0 = y[0], y1 = y[1], y2 = y[2], y3 = y[3];
              const u64 m0 = __builtin_amdgcn_ballot_w64(!(y0 <= 0.f)), m1 = __builtin_amdgcn_ballot_w64(!(y1 <= 0.f));
              const u64 m2 = __builtin_amdgcn_ballot_w64(!(y2 <= 0.f)), m3 = __builtin_amdgcn_ballot_w64(!(y3 <= 0.f));
              unsigned w1 = wp + __builtin_popcountll(m0) * 4; asm("" : "+s"(w1));
              unsigned w2 = w1 + __builtin_popcountll(m1) * 4; asm("" : "+s"(w2));
              unsigned w3 = w2 + __builtin_popcountll(m2) * 4; asm("" : "+s"(w3));
              const unsigned a0 = mbcnt(m0, 0) * 4 + wp, a1 = mbcnt(m1, 0) * 4 + w1, a2 = mbcnt(m2, 0) * 4 + w2, a3 = mbcnt(m3, 0) * 4 + w3;
              wp = w3 + __builtin_popcountll(m3) * 4; asm("" : "+s"(wp));
              asm volatile("s_mov_b64 exec, %0\n\tds_write_b32 %4, %8\n\ts_mov_b64 exec, %1\n\tds_write_b32 %5, %9\n\t"
                           "s_mov_b64 exec, %2\n\tds_write_b32 %6, %10\n\ts_mov_b64 exec, %3\n\tds_write_b32 %7, %11\n\ts_mov_b64 exec, -1"
                           :: "s"(m0), "s"(m1), "s"(m2), "s"(m3), "v"(a0), "v"(a1), "v"(a2), "v"(a3), "v"(y0), "v"(y1), "v"(y2), "v"(y3) : "memory");
              const int ri = (mi * 2 + nj) * 8;
              hdr = wrlane((unsigned)m0, ri, hdr);     hdr = wrlane((unsigned)(m0 >> 32), ri + 1, hdr);
              hdr = wrlane((unsigned)m1, ri + 2, hdr); hdr = wrlane((unsigned)(m1 >> 32), ri + 3, hdr);
              hdr = wrlane((unsigned)m2, ri + 4, hdr); hdr = wrlane((unsigned)(m2 >> 32), ri + 5, hdr);
              hdr = wrlane((unsigned)m3, ri + 6, hdr); hdr = wrlane((unsigned)(m3 >> 32), ri + 7, hdr);
            }
          const unsigned cnt = (wp - msg_base) >> 2;
          const unsigned z = (unsigned)__builtin_amdgcn_ballot_w64(hdr != 0);
          hdr = wrlane((z & 0xffffu) | 0x10000u, 32, hdr);
          hdr = wrlane((z >> 16) | 0x10000u, 33, hdr);
          if (lane < HDR_DW + 3) msg[lane < HDR_DW ? lane : cnt + lane - HDR_DW] = hdr;
          const int n16 = (cnt + 3) >> 2;
          for (int c = lane; c < n16; c += 64) {
            v4u v = *(LDS v4u*)(msg + c * 4);
            if (thr == 1) wait_vm<0>(); else if (thr == 2) wait_vm<1>();
            __builtin_amdgcn_raw_buffer_store_b128(v, rd_, h * SLOT + c * 16, 0, AUX_SYS);
          }
          if (ti == 0 && h == 0) STAMP(2);
        }
      }
      td = td1;
      if (!(td & 1024)) break;
    }
  }
  const int rj = wv8 ^ 4;
  call = __builtin_amdgcn_readfirstlane(call);
  if (rj < NR - 1) {
    const int r = lane & 15, g = lane >> 4;
    const int src = (p.rank + 1 + rj) & 7;
    __amdgpu_buffer_rsrc_t rs = __builtin_amdgcn_make_buffer_rsrc(
        (void*)(p.myws + WS_STAGE + ((call & 1) * NR + src) * PAIR_BYTES + b * SLOT), 0, SLOT, 0x00020000);
    __amdgpu_buffer_rsrc_t ro = __builtin_amdgcn_make_buffer_rsrc((void*)p.out, 0, NR * MLOC * NSH * 4, 0x00020000);
    unsigned* tmo = (unsigned*)(p.myws + WS_TMO);
    const u64 ts = __builtin_amdgcn_s_memrealtime();
    bool dead = false;
    auto expired = [&]() {
      if (__hip_atomic_load(tmo, __ATOMIC_RELAXED, __HIP_MEMORY_SCOPE_AGENT) == call ||
          __builtin_amdgcn_s_memrealtime() - ts > DEADLINE_TICKS) {
        __hip_atomic_store(tmo, call, __ATOMIC_RELAXED, __HIP_MEMORY_SCOPE_AGENT);
        return true;
      }
      return false;
    };
    unsigned hdr;
    const int hoff = lane < HDR_DW ? lane * 4 : SLOT;
    for (unsigned spin = 1;; ++spin) {
      hdr = __builtin_amdgcn_raw_buffer_load_b32(rs, hoff, 0, AUX_SYS);
      const u64 nz = __builtin_amdgcn_ballot_w64(hdr != 0);
      const unsigned z = ((unsigned)__builtin_amdgcn_readlane(hdr, 32) & 0xffffu) | ((unsigned)__builtin_amdgcn_readlane(hdr, 33) << 16);
      if ((unsigned)(nz >> 32) == 3u && (unsigned)nz == z) break;
      if ((spin & 255) == 0 && expired()) { dead = true; break; }
      __builtin_amdgcn_s_sleep(4);
    }
    STAMP(5);
    unsigned off[16];
    unsigned cnt4 = HDR_DW * 4;
#pragma unroll
    for (int ri = 0; ri < 16; ++ri) {
      const u64 m = (u64)(unsigned)__builtin_amdgcn_readlane(hdr, 2 * ri) | ((u64)(unsigned)__builtin_amdgcn_readlane(hdr, 2 * ri + 1) << 32);
      const unsigned o = mbcnt(m, 0) * 4 + cnt4;
      off[ri] = __builtin_amdgcn_inverse_ballot_w64(m) ? o : SLOT;
      cnt4 += __builtin_popcountll(m) * 4;
      asm("" : "+s"(cnt4));
    }
    unsigned v[16];
    for (unsigned spin = 1; !dead; ++spin) {
      bool miss = false;
#pragma unroll
      for (int ri = 0; ri < 16; ++ri) v[ri] = __builtin_amdgcn_raw_buffer_load_b32(rs, off[ri], 0, AUX_SYS);
#pragma unroll
      for (int ri = 0; ri < 16; ++ri) miss |= (v[ri] == 0) & (off[ri] != SLOT);
      if (!__builtin_amdgcn_ballot_w64(miss)) break;
      if ((spin & 63) == 0 && expired()) { dead = true; break; }
      __builtin_amdgcn_s_sleep(1);
    }
    if (dead) {
#pragma unroll
      for (int ri = 0; ri < 16; ++ri) v[ri] = 0x7fc00000u;
    }
    const int obase = ((src * MLOC + (b >> 4) * 32 + r) * NSH + (b & 15) * 32 + g * 4) * 4;
#pragma unroll
    for (int mi = 0; mi < 2; ++mi)
#pragma unroll
      for (int nj = 0; nj < 2; ++nj) {
        const int ri = (mi * 2 + nj) * 4;
        v4u y = {v[ri], v[ri + 1], v[ri + 2], v[ri + 3]};
        __builtin_amdgcn_raw_buffer_store_b128(y, ro, obase + mi * 16 * NSH * 4 + nj * 64, 0, AUX_WT);
      }
    for (unsigned c = lane * 16; c < cnt4; c += 1024)
      __builtin_amdgcn_raw_buffer_store_b128((v4u){0u, 0u, 0u, 0u}, rs, c, 0, AUX_SYS);
  } else if (lane == 0) {
    __hip_atomic_store((unsigned*)(p.myws + WS_CTR) + b, call, __ATOMIC_RELAXED, __HIP_MEMORY_SCOPE_AGENT);
  }
}

}

static int envi(const char*, int d) { return d; }

static void build_table(Params& p) {
  const int ne = envi("NE", 0), nl1 = envi("NL1", 14), D = envi("DLY", 400), stag = envi("STAG", 0), pf = envi("PF", 0);
  static const int rowslot[8] = {0, 1, 4, 5, 2, 3, 6, 7};
  for (int g = 0; g < 8; ++g) {
    int tl[28], n = 0;
    for (int tm = 0; tm < 8; ++tm)
      for (int q = 0; q < 28; ++q) { int t = g * 28 + q; if (rowslot[t & 7] == tm) tl[n++] = t; }
    auto desc = [&](int strip_global, int tm, int delay) {
      return (unsigned)((strip_global * 2) | (tm << 6) | 1024 | ((stag ? ((tm + g) & 7) : 0) << 11) | (delay << 16));
    };
    auto rdesc = [&](int t, int delay) { int dest = (p.rank + 1 + t / 32) & 7; return desc(dest * 4 + ((t >> 3) & 3), rowslot[t & 7], delay); };
    unsigned h0 = 0; int ns = 0, maxtm = 0;
    for (int i = 0; i < ne + nl1 && i < 28; ++i) {
      unsigned s = (rdesc(tl[i], 0) & 63) >> 1; bool seen = false;
      for (int k = 0; k < ns; ++k) seen |= ((h0 >> (5 * k)) & 31) == s;
      if (!seen && ns < 5) h0 |= s << (5 * ns++);
      if (rowslot[tl[i] & 7] > maxtm) maxtm = rowslot[tl[i] & 7];
    }
    h0 |= (unsigned)(pf ? ns : 0) << 25;
    p.grp[g] = h0;
    unsigned rest[32]; int nr = 0;
    for (int i = ne; i < 28; ++i) rest[nr++] = rdesc(tl[i], 0);
    for (int i = 0; i < 4; ++i) rest[nr++] = desc(p.rank * 4 + (g >> 1), (g & 1) * 4 + i, 0);
    const int nsingle = 32 - 2 * ne, nlate = nsingle - nl1;
    for (int idx = 0; idx < 32; ++idx) {
      unsigned* t = p.tab[idx * 8 + g];
      t[1] = 0;
      if (idx < 2 * ne) {
        t[0] = rdesc(tl[idx >> 1], 0) | 512 | (idx & 1);
        if (!(idx & 1)) t[1] = rest[nsingle + (idx >> 1)];
      } else {
        const int i = idx - 2 * ne;
        t[0] = rest[i];
        if (i >= nl1 && D > 0) { t[0] |= (unsigned)D << 16; t[1] = (unsigned)((i - nl1) | (nlate << 6) | ((maxtm + 1) << 12)) << 11; }
      }
    }
  }
}

extern "C" void kernel_launch(int rank, int nranks, ncclComm_t comm, hipStream_t stream,
                              void* const* d_in, const int* in_sizes, int n_in,
                              void* d_out, int out_size, const int* in_dtypes, int out_dtype,
                              void* const* d_out_all, void* const* d_ws_all, size_t ws_bytes) {
  if (nranks != NR || (unsigned)rank >= (unsigned)NR || ws_bytes < WS_NEED) return;
  Params p;
  p.x = (const unsigned char*)d_in[0];
  p.w = (const unsigned char*)d_in[1];
  p.sx = (const float*)d_in[2];
  p.sw = (const float*)d_in[3];
  p.out = (float*)d_out;
  for (int i = 0; i < NR; ++i) p.ws[i] = (unsigned char*)d_ws_all[i];
  p.myws = p.ws[rank];
  p.rank = rank;
  p.thr = envi("THR", 2);
  p.pfstart = envi("PFS", 4);
  p.pad = 0;
  build_table(p);
  gemm_a2a<<<NWG, 512, 0, stream>>>(p);
  HIP_CHECK(hipGetLastError());
}
```

```cpp
#include <hip/hip_runtime.h>
#include <rccl/rccl.h>
#include <cstdio>
#include <cstdlib>

#define HIP_CHECK(expr) do { hipError_t _e = (expr); if (_e != hipSuccess) { \
    fprintf(stderr, "HIP error %s at %s:%d\n", hipGetErrorString(_e), __FILE__, __LINE__); \
    return; } } while (0)

namespace {

typedef int v2i __attribute__((ext_vector_type(2)));
typedef int v4i __attribute__((ext_vector_type(4)));
typedef unsigned v4u __attribute__((ext_vector_type(4)));
typedef int v8i __attribute__((ext_vector_type(8)));
typedef float v4f __attribute__((ext_vector_type(4)));
typedef unsigned long long u64;
#define LDS __attribute__((address_space(3)))

constexpr int NR = 8;
constexpr int MLOC = 512, NN = 4096, KK = 4096, NSH = 512;
constexpr int STAGES = 5;
constexpr int XB = 8192, SB = 24576;
constexpr int NS = KK / 128;
constexpr int NWG = 256;
#define DEADLINE 100000000ull
constexpr u64 DEADLINE_TICKS = DEADLINE;

constexpr int HDR_B = 136;
constexpr int SLOT = 4352;
constexpr int UNITS = 256;
constexpr int PAIR_BYTES = UNITS * SLOT;
constexpr int MSG_OFF = SB * STAGES;
constexpr int MISC_OFF = MSG_OFF + 8 * SLOT;
constexpr int LDS_BYTES = MISC_OFF + 1024;
static_assert(LDS_BYTES <= 163840, "LDS");
constexpr unsigned WS_CTR = 0;
constexpr unsigned WS_TMO = 4096;
constexpr unsigned WS_STAMP = 65536;
constexpr unsigned WS_STAGE = 1u << 20;
constexpr unsigned WS_XCH = WS_STAGE + 2u * NR * PAIR_BYTES;
constexpr unsigned WS_NEED = WS_XCH + NWG * 32768u;
constexpr int AUX_SYS = 17;
constexpr int AUX_WT = 16;

struct Params {
  const unsigned char* x;
  const unsigned char* w;
  unsigned char* myws;
  float* out;
  const float* sx;
  const float* sw;
  int rank, thr, na, cs;
  int ps, te, tl, xa;
  unsigned char* ws[NR];
  unsigned tab[NWG][2];
};

template <int N_> __device__ __forceinline__ void wait_vm() { asm volatile("s_waitcnt vmcnt(%0)" :: "i"(N_) : "memory"); }
__device__ __forceinline__ void dma16(__amdgpu_buffer_rsrc_t rs, unsigned lds_dst, int voff, int soff) {
  asm volatile("s_mov_b32 m0, %0\n\ts_nop 0\n\tbuffer_load_dwordx4 %1, %2, %3 offen lds"
               :: "s"(lds_dst), "v"(voff), "s"(rs), "s"(soff) : "memory");
}
__device__ __forceinline__ void dma16_sys(__amdgpu_buffer_rsrc_t rs, unsigned lds_dst, int voff, int soff) {
  asm volatile("s_mov_b32 m0, %0\n\ts_nop 0\n\tbuffer_load_dwordx4 %1, %2, %3 offen sc0 sc1 lds"
               :: "s"(lds_dst), "v"(voff), "s"(rs), "s"(soff) : "memory");
}
#define SCHED() __builtin_amdgcn_sched_barrier(0)
#define BAR() do { SCHED(); asm volatile("s_barrier" ::: "memory"); SCHED(); } while (0)
#define LBAR() do { SCHED(); __builtin_amdgcn_s_waitcnt(0xC07F); asm volatile("s_barrier" ::: "memory"); SCHED(); } while (0)
__device__ __forceinline__ unsigned now32() { return (unsigned)__builtin_amdgcn_s_memrealtime(); }
__device__ __forceinline__ unsigned wave_scan(unsigned x) {
  x += __builtin_amdgcn_update_dpp(0u, x, 0x111, 0xf, 0xf, true);
  x += __builtin_amdgcn_update_dpp(0u, x, 0x112, 0xf, 0xf, true);
  x += __builtin_amdgcn_update_dpp(0u, x, 0x114, 0xf, 0xf, true);
  x += __builtin_amdgcn_update_dpp(0u, x, 0x118, 0xf, 0xf, true);
  x += __builtin_amdgcn_update_dpp(0u, x, 0x142, 0xa, 0xf, false);
  x += __builtin_amdgcn_update_dpp(0u, x, 0x143, 0xc, 0xf, false);
  return x;
}

struct Frags { v8i xf[2]; v8i wf[4]; };

#define STAMP(i)

__global__ void __launch_bounds__(512, 2) gemm_a2a(Params p) {
  __shared__ __attribute__((aligned(1024))) unsigned char smem[LDS_BYTES];
  const int tid = threadIdx.x, lane = tid & 63;
  const int wv8 = __builtin_amdgcn_readfirstlane(tid >> 6);
  const int wv = wv8 & 3;
  const int b = blockIdx.x;
  const unsigned smem_base = (unsigned)(size_t)(LDS unsigned char*)smem;
  const unsigned tbegin = now32();
  const unsigned td = p.tab[b][0], td2 = p.tab[b][1];
  const unsigned kind = td >> 30;
  const int n0 = kind == 2 ? NS - p.na : 0;
  const int nsteps = n0 + (kind == 1 ? p.na : NS);
  const int r = lane & 15, g = lane >> 4;
  unsigned* tmo = (unsigned*)(p.myws + WS_TMO);
  unsigned call, rbuf;

  if (wv8 >= 4) {
    __amdgpu_buffer_rsrc_t rx = __builtin_amdgcn_make_buffer_rsrc((void*)p.x, 0, MLOC * KK, 0x00020000);
    __amdgpu_buffer_rsrc_t rw = __builtin_amdgcn_make_buffer_rsrc((void*)p.w, 0, KK * NN, 0x00020000);
    const int l3 = lane >> 3, l7 = lane & 7;
    const int c0 = (l7 ^ (l3 >> 1)) << 4;
    const int xo0 = (wv * 16 + l3) * KK + c0;
    const int xo1 = (xo0 + 8 * KK) ^ 64;
    const int wo0 = (wv * 32 + l3) * NN + c0, wo1 = wo0 + 8 * NN;
    const int wo2 = (wo0 + 16 * NN) ^ 64, wo3 = wo2 + 8 * NN;
    unsigned cur = kind == 2 ? td2 : td;
    int klo = kind == 2 ? p.na : 0;
    const int khi = kind == 1 ? p.na : NS;
    int kc = klo + ((cur >> 8) & 31), left = khi - klo;
    unsigned dst = smem_base + wv * 2048;
    auto issue = [&]() {
      const int so = (cur & 31) * 128 + kc * 128 * NN, sx = ((cur >> 5) & 7) * 64 * KK + kc * 128;
      const unsigned dw = dst + XB + wv * 2048;
      dma16(rw, dw, wo0, so); dma16(rw, dw + 1024, wo1, so); dma16(rw, dw + 2048, wo2, so); dma16(rw, dw + 3072, wo3, so);
      dma16(rx, dst, xo0, sx); dma16(rx, dst + 1024, xo1, sx);
      if (++kc == khi) kc = klo;
      if (--left == 0) { cur = td; klo = 0; kc = (td >> 8) & 31; left = NS; }
      dst += SB; if (dst >= smem_base + STAGES * SB) dst -= STAGES * SB;
    };
    const unsigned delay = (td >> 16) & 0x3fff;
    if (delay) {
      const unsigned tstart = now32();
      while (now32() - tstart < delay) __builtin_amdgcn_s_sleep(2);
    }
    STAMP(1);
    issue(); issue();
    wait_vm<6>();
    STAMP(2);
    BAR();
#pragma nounroll
    for (int s = 2; s < STAGES; ++s) issue();
#pragma nounroll
    for (int kt = 0; kt < nsteps - STAGES; ++kt) {
      wait_vm<6 * (STAGES - 2)>();
      BAR();
      issue();
    }
    static_assert(STAGES == 5, "tail");
    wait_vm<18>(); BAR();
    wait_vm<12>(); BAR();
    wait_vm<6>(); BAR();
    wait_vm<0>();
    STAMP(3);
    BAR();
    call = *(volatile LDS unsigned*)(smem + MISC_OFF);
    rbuf = dst + wv * (5120 - 2048);
  } else {
    const int dest = (td >> 2) & 7, tm = (td >> 5) & 7, tn = td & 3;
    call = __builtin_amdgcn_readfirstlane(__hip_atomic_load((const unsigned*)(p.myws + WS_CTR) + b, __ATOMIC_RELAXED, __HIP_MEMORY_SCOPE_AGENT)) + 1;
    const float scale = p.sx[0] * p.sw[0];
    if (tid == 0) *(volatile LDS unsigned*)(smem + MISC_OFF) = call;
    const int wm = wv & 1, wn = wv >> 1;
    rbuf = smem_base + MSG_OFF + wv * 2 * SLOT;
    const int ucol0 = 4 * tn + 2 * wn, urow = tm * 2 + wm;
    unsigned char* dptr = p.ws[dest] + WS_STAGE + ((call & 1) * NR + p.rank) * PAIR_BYTES + (urow * 16 + ucol0) * SLOT;
    int thr = p.thr;
    asm volatile("" : "+s"(dptr), "+s"(thr));
    int wa[4];
    {
      const int q = r >> 1, pp = r & 1, fl = ((q >> 1) & 3) | ((g & 1) << 2);
#pragma unroll
      for (int ni = 0; ni < 4; ++ni) wa[ni] = XB + (16 * g + q) * 128 + (((wn * 4 + ni) ^ fl) << 4) + pp * 8;
    }
    const int xa0 = (wm * 32 + r) * 128 + ((g ^ (r >> 1)) << 4);
    const int xa1 = xa0 ^ 64;
    auto rd = [&](Frags& f, int sb) {
#pragma unroll
      for (int mi = 0; mi < 2; ++mi) {
        v4i a = *(LDS v4i*)(smem + sb + xa0 + mi * 2048);
        v4i c = *(LDS v4i*)(smem + sb + xa1 + mi * 2048);
        f.xf[mi] = __builtin_shufflevector(a, c, 0, 1, 2, 3, 4, 5, 6, 7);
      }
#pragma unroll
      for (int ni = 0; ni < 4; ++ni) {
#pragma unroll
        for (int j = 0; j < 4; ++j) {
          v2i t = __builtin_amdgcn_ds_read_tr8_b64_v2i32((LDS v2i*)(smem + sb + wa[ni] + (j & 1) * 1024 + (j >> 1) * 8192));
          f.wf[ni][2 * j] = t[0]; f.wf[ni][2 * j + 1] = t[1];
        }
      }
    };
    v4f acc[8];
#pragma unroll
    for (int i = 0; i < 8; ++i) acc[i] = (v4f){0.f, 0.f, 0.f, 0.f};
    auto mma = [&](const Frags& f) {
#pragma unroll
      for (int ni = 0; ni < 4; ++ni)
#pragma unroll
        for (int mi = 0; mi < 2; ++mi) {
          const int i = (ni >> 1) * 4 + mi * 2 + (ni & 1);
          acc[i] = __builtin_amdgcn_mfma_scale_f32_16x16x128_f8f6f4(f.wf[ni], f.xf[mi], acc[i],  1,  0, 0, 0, 0, 0);
        }
    };
    auto interleave = [&]() {
#pragma unroll
      for (int i = 0; i < 8; ++i) {
        __builtin_amdgcn_sched_group_barrier(0x008, 1, 0);
        __builtin_amdgcn_sched_group_barrier(0x002, 2, 0);
        __builtin_amdgcn_sched_group_barrier(0x100, 4, 0);
      }
    };
    const unsigned tag = call & 1;
    __amdgpu_buffer_rsrc_t rxch = __builtin_amdgcn_make_buffer_rsrc(
        (void*)(p.myws + WS_XCH + (kind == 2 ? (td2 >> 16) & 255 : b) * 32768u + wv * 8192), 0, 8192, 0x00020000);
    Frags f0, f1;
    LBAR();
    rd(f0, 0);
    int sb = SB;
#pragma nounroll
    for (int kt = 0; kt + 2 < nsteps; kt += 2) {
      LBAR();
      rd(f1, sb); mma(f0); interleave();
      sb += SB; if (sb == STAGES * SB) sb = 0;
      LBAR();
      rd(f0, sb); mma(f1); interleave();
      sb += SB; if (sb == STAGES * SB) sb = 0;
      if (kt + 2 == n0) {
        SCHED();
#pragma unroll
        for (int i = 0; i < 8; ++i) {
          const v4u y = (__builtin_bit_cast(v4u, acc[i]) & ~1u) | tag;
          __builtin_amdgcn_raw_buffer_store_b128(y, rxch, i * 1024 + lane * 16, 0, AUX_WT);
          acc[i] = (v4f){0.f, 0.f, 0.f, 0.f};
        }
        STAMP(4);
      }
    }
    v4u part[8];
    if (kind == 1) {
#pragma unroll
      for (int i = 0; i < 8; ++i) part[i] = __builtin_amdgcn_raw_buffer_load_b128(rxch, i * 1024 + lane * 16, 0, AUX_WT);
    }
    LBAR();
    rd(f1, sb); mma(f0); interleave();
    SCHED();
    mma(f1);
    SCHED();
    if (kind == 1) {
      STAMP(4);
      const u64 ts = __builtin_amdgcn_s_memrealtime();
      const unsigned flip = tag ? ~0u : 0u;
      for (unsigned spin = 1;; ++spin) {
        v4u x = part[0] ^ flip;
#pragma unroll
        for (int i = 1; i < 8; ++i) x |= part[i] ^ flip;
        if (!__builtin_amdgcn_ballot_w64(((x[0] | x[1] | x[2] | x[3]) & 1u) != 0)) break;
        bool dead = false;
        while (((__builtin_amdgcn_readfirstlane(__builtin_amdgcn_raw_buffer_load_b32(rxch, 8188, 0, AUX_WT)) ^ tag) & 1u) || (spin & 7) == 0) {
          if ((++spin & 7) == 1 && (__hip_atomic_load(tmo, __ATOMIC_RELAXED, __HIP_MEMORY_SCOPE_AGENT) == call ||
                                    __builtin_amdgcn_s_memrealtime() - ts > DEADLINE_TICKS)) {
            __hip_atomic_store(tmo, call, __ATOMIC_RELAXED, __HIP_MEMORY_SCOPE_AGENT);
            dead = true;
            break;
          }
          __builtin_amdgcn_s_sleep(2);
        }
        if (dead) {
#pragma unroll
          for (int i = 0; i < 8; ++i) part[i] = (v4u){0x7fc00000u, 0x7fc00000u, 0x7fc00000u, 0x7fc00000u};
          break;
        }
#pragma unroll
        for (int i = 0; i < 8; ++i) part[i] = __builtin_amdgcn_raw_buffer_load_b128(rxch, i * 1024 + lane * 16, 0, AUX_WT);
      }
#pragma unroll
      for (int i = 0; i < 8; ++i) acc[i] += __builtin_bit_cast(v4f, part[i] & ~1u);
    }
    STAMP(1);
    if (dest == p.rank) {
      __amdgpu_buffer_rsrc_t ro = __builtin_amdgcn_make_buffer_rsrc((void*)p.out, 0, NR * MLOC * NSH * 4, 0x00020000);
      const int obase = ((p.rank * MLOC + urow * 32 + r) * NSH + ucol0 * 32 + g * 4) * 4;
#pragma unroll
      for (int i = 0; i < 8; ++i) {
        v4f y = acc[i] * scale;
#pragma unroll
        for (int j = 0; j < 4; ++j) y[j] = (y[j] <= 0.f) ? 0.f : y[j];
        __builtin_amdgcn_raw_buffer_store_b128(__builtin_bit_cast(v4u, y), ro, obase + ((i >> 1) & 1) * 16 * NSH * 4 + (i >> 2) * 128 + (i & 1) * 64, 0, AUX_WT);
      }
    } else {
      __amdgpu_buffer_rsrc_t rd_ = __builtin_amdgcn_make_buffer_rsrc((void*)dptr, 0, 2 * SLOT, 0x00020000);
      auto send_chunk = [&](int h, int sent, int total) {
        const int c = sent + lane;
        v4u v = *(LDS v4u*)(smem + MSG_OFF + (wv * 2 + h) * SLOT + c * 16);
        if (total - sent >= 24) { if (thr == 1) wait_vm<0>(); else if (thr == 2) wait_vm<1>(); }
        __builtin_amdgcn_raw_buffer_store_b128(v, rd_, c < total ? h * SLOT + c * 16 : 2 * SLOT, 0, AUX_SYS);
      };
      auto pack = [&](const v4f* a4, unsigned msg) -> int {
        float y[16]; u64 m[16];
        unsigned hm = 0;
#pragma unroll
        for (int i = 0; i < 4; ++i) {
          const v4f t = a4[i] * scale;
#pragma unroll
          for (int j = 0; j < 4; ++j) {
            y[i * 4 + j] = t[j];
            m[i * 4 + j] = __builtin_amdgcn_ballot_w64(!(t[j] <= 0.f));
          }
        }
#define AC(n) "v_addc_co_u32 %0, vcc, %0, %0, %" #n "\n\t"
        asm("s_nop 1\n\t" AC(1) AC(2) AC(3) AC(4) AC(5) AC(6) AC(7) AC(8) AC(9) AC(10) AC(11) AC(12) AC(13) AC(14) AC(15) AC(16) "s_nop 1"
            : "+v"(hm) : "s"(m[0]), "s"(m[1]), "s"(m[2]), "s"(m[3]), "s"(m[4]), "s"(m[5]), "s"(m[6]), "s"(m[7]),
              "s"(m[8]), "s"(m[9]), "s"(m[10]), "s"(m[11]), "s"(m[12]), "s"(m[13]), "s"(m[14]), "s"(m[15]) : "vcc");
#undef AC
        const unsigned cnt = __builtin_popcount(hm);
        const unsigned incl = wave_scan(cnt);
        const unsigned nval = __builtin_amdgcn_readlane(incl, 63);
        unsigned ad = msg + (incl - cnt) * 4;
#define WR(mm, yy) "s_mov_b64 exec, %" #mm "\n\tds_write_b32 %0, %" #yy " offset:136\n\tv_add_u32 %0, 4, %0\n\t"
#pragma unroll
        for (int i = 0; i < 16; i += 8)
          asm volatile(WR(1, 9) WR(2, 10) WR(3, 11) WR(4, 12) WR(5, 13) WR(6, 14) WR(7, 15) WR(8, 16) "s_mov_b64 exec, -1"
                       : "+v"(ad) : "s"(m[i]), "s"(m[i + 1]), "s"(m[i + 2]), "s"(m[i + 3]), "s"(m[i + 4]), "s"(m[i + 5]), "s"(m[i + 6]), "s"(m[i + 7]),
                         "v"(y[i]), "v"(y[i + 1]), "v"(y[i + 2]), "v"(y[i + 3]), "v"(y[i + 4]), "v"(y[i + 5]), "v"(y[i + 6]), "v"(y[i + 7]) : "memory");
#undef WR
        static_assert(HDR_B == 136, "offset above");
        *(LDS unsigned short*)(size_t)(msg + (lane & 31) * 4 + (lane >> 5) * 2) = (unsigned short)hm;
        const u64 nz = __builtin_amdgcn_ballot_w64(hm != 0);
        const unsigned z = (unsigned)nz | (unsigned)(nz >> 32);
        const unsigned pa = lane < 2 ? lane * 4 : nval * 4 + min(lane, 4) * 4;
        *(LDS unsigned*)(size_t)(msg + 128 + pa) = lane < 2 ? (((z >> (lane * 16)) & 0xffffu) | 0x10000u) : 0u;
        return (HDR_B + nval * 4 + 15) >> 4;
      };
      const int tot0 = pack(acc, rbuf);
      send_chunk(0, 0, tot0);
      STAMP(2);
      const int tot1 = pack(acc + 4, rbuf + SLOT);
#pragma nounroll
      for (int c = 64; c < tot0; c += 64) send_chunk(0, c, tot0);
#pragma nounroll
      for (int c = 0; c < tot1; c += 64) send_chunk(1, c, tot1);
      STAMP(3);
    }
    if (tid == 0)
      __hip_atomic_store((unsigned*)(p.myws + WS_CTR) + b, call, __ATOMIC_RELAXED, __HIP_MEMORY_SCOPE_AGENT);
  }
  call = __builtin_amdgcn_readfirstlane(call);
  rbuf = __builtin_amdgcn_readfirstlane(rbuf);
#pragma nounroll
  for (int late = wv8 >= 4 ? 0 : 1; late < (b < 112 ? 2 : (b < 224 ? 1 : 0)); ++late) {
    const int idx = late ? b * 8 + wv8 : b * 4 + wv;
    const int q = idx / 7, src = (p.rank + 1 + idx - q * 7) & 7;
    const int un = p.cs ? (q >> 3) * 16 + (q & 7) + late * 8 : q + late * 128;
    __amdgpu_buffer_rsrc_t rs = __builtin_amdgcn_make_buffer_rsrc(
        (void*)(p.myws + WS_STAGE + ((call & 1) * NR + src) * PAIR_BYTES + un * SLOT), 0, SLOT, 0x00020000);
    __amdgpu_buffer_rsrc_t ro = __builtin_amdgcn_make_buffer_rsrc((void*)p.out, 0, NR * MLOC * NSH * 4, 0x00020000);
    const u64 ts = __builtin_amdgcn_s_memrealtime();
    bool dead = false;
    auto expired = [&]() {
      if (__hip_atomic_load(tmo, __ATOMIC_RELAXED, __HIP_MEMORY_SCOPE_AGENT) == call ||
          __builtin_amdgcn_s_memrealtime() - ts > DEADLINE_TICKS) {
        __hip_atomic_store(tmo, call, __ATOMIC_RELAXED, __HIP_MEMORY_SCOPE_AGENT);
        return true;
      }
      return false;
    };
    while (now32() - tbegin < (unsigned)(late ? p.tl : p.te)) __builtin_amdgcn_s_sleep(8);
    unsigned hdr;
    const int hoff = lane < 34 ? lane * 4 : SLOT;
    for (unsigned spin = 1;; ++spin) {
      hdr = __builtin_amdgcn_raw_buffer_load_b32(rs, hoff, 0, AUX_SYS);
      const u64 nz = __builtin_amdgcn_ballot_w64(hdr != 0);
      const unsigned z = ((unsigned)__builtin_amdgcn_readlane(hdr, 32) & 0xffffu) | ((unsigned)__builtin_amdgcn_readlane(hdr, 33) << 16);
      if ((unsigned)(nz >> 32) == 3u && (unsigned)nz == z) break;
      if ((spin & 255) == 0 && expired()) { dead = true; hdr = 0; break; }
      for (int i = 0; i < p.ps; ++i) __builtin_amdgcn_s_sleep(4);
    }
    STAMP(5);
    dma16_sys(rs, rbuf, lane * 16, 0); dma16_sys(rs, rbuf + 1024, lane * 16, 1024); dma16_sys(rs, rbuf + 2048, lane * 16, 2048);
    const unsigned lo = __builtin_amdgcn_permlane32_swap(hdr, hdr, false, false)[0];
    const unsigned hm = lane < 32 ? (lo & 0xffffu) : (lo >> 16);
    const unsigned cnt = __builtin_popcount(hm);
    const unsigned incl = wave_scan(cnt);
    const unsigned endb = HDR_B + __builtin_amdgcn_readlane(incl, 63) * 4;
    const unsigned first = HDR_B + (incl - cnt) * 4;
    for (unsigned c = 3072; c <= endb; c += 1024) dma16_sys(rs, rbuf + c, lane * 16, c);
    unsigned ad[16];
    {
      unsigned cur = rbuf + first;
#pragma unroll
      for (int i = 0; i < 16; ++i) {
        const bool keep = (hm >> (15 - i)) & 1;
        ad[i] = keep ? cur : rbuf + endb;
        cur += keep ? 4 : 0;
      }
    }
    unsigned v[16];
    for (unsigned spin = 1;; ++spin) {
      wait_vm<0>();
      unsigned got = 0;
#pragma unroll
      for (int i = 0; i < 16; ++i) v[i] = *(volatile LDS unsigned*)(size_t)ad[i];
#pragma unroll
      for (int i = 0; i < 16; ++i) got += min(v[i], 1u);
      const u64 miss = __builtin_amdgcn_ballot_w64(got != cnt);
      if (!miss) break;
      if ((spin & 63) == 0 && expired()) { dead = true; break; }
      __builtin_amdgcn_s_sleep(2);
      for (unsigned c = __builtin_amdgcn_readlane(first, __builtin_ctzll(miss)) & ~1023u; c < endb; c += 1024) dma16_sys(rs, rbuf + c, lane * 16, c);
    }
    STAMP(7);
    if (dead) {
#pragma unroll
      for (int i = 0; i < 16; ++i) v[i] = 0x7fc00000u;
    }
    const int obase = ((src * MLOC + (un >> 4) * 32 + r) * NSH + (un & 15) * 32 + g * 4) * 4;
#pragma unroll
    for (int i = 0; i < 4; ++i) {
      v4u y = {v[i * 4], v[i * 4 + 1], v[i * 4 + 2], v[i * 4 + 3]};
      __builtin_amdgcn_raw_buffer_store_b128(y, ro, obase + (i >> 1) * 16 * NSH * 4 + (i & 1) * 64, 0, AUX_WT);
    }
    for (unsigned c = lane * 16; c < endb; c += 1024)
      __builtin_amdgcn_raw_buffer_store_b128((v4u){0u, 0u, 0u, 0u}, rs, c, 0, AUX_SYS);
    if (!late) STAMP(4);
  }
}

}

static int envi(const char*, int d) { return d; }

static void build_table(Params& p) {
  const int D = envi("DLY", 300), stag = envi("STAG", 1);
  static const int rowslot[8] = {0, 1, 4, 5, 2, 3, 6, 7};
  const int na = p.na;
  for (int g = 0; g < 8; ++g) {
    unsigned tl[32]; int n = 0;
    auto desc = [&](int strip_global, int tm) { return (unsigned)(strip_global | (tm << 5)); };
    if (p.cs) {
      for (int late = 0; late < 2; ++late)
        for (int q = 0; q < 14; ++q) { int e = g * 14 + q; tl[n++] = desc(((p.rank + 1 + e / 16) & 7) * 4 + late * 2 + ((e >> 3) & 1), e & 7); }
    } else {
      for (int tm = 0; tm < 8; ++tm)
        for (int q = 0; q < 28; ++q) {
          int t = g * 28 + q;
          if (rowslot[t & 7] == tm) tl[n++] = desc(((p.rank + 1 + t / 32) & 7) * 4 + ((t >> 3) & 3), tm);
        }
    }
    for (int i = 0; i < 4; ++i) tl[n++] = desc(p.rank * 4 + (g >> 1), (g & 1) * 4 + i);
    auto rot = [&](unsigned t, int len, int ways) { return stag ? (unsigned)((((t >> 5) + g) & (ways - 1)) * len / ways) << 8 : 0u; };
    const int ways = p.cs ? 8 : 4;
    for (int i = 0; i < 32; ++i) {
      unsigned* t = p.tab[i * 8 + g];
      t[1] = 0;
      if (i >= 28) t[0] = tl[i] | rot(tl[i], 32, 4) | ((unsigned)D << 16);
      else if (na >= NS) t[0] = tl[i] | rot(tl[i], 32, ways);
      else if (i < 14) t[0] = tl[i] | rot(tl[i], na, ways) | (1u << 30);
      else { t[0] = tl[i] | rot(tl[i], 32, ways) | (2u << 30); t[1] = tl[i - 14] | rot(tl[i - 14], NS - na, ways) | ((unsigned)((i - 14) * 8 + g) << 16); }
    }
  }
}

extern "C" void kernel_launch(int rank, int nranks, ncclComm_t comm, hipStream_t stream,
                              void* const* d_in, const int* in_sizes, int n_in,
                              void* d_out, int out_size, const int* in_dtypes, int out_dtype,
                              void* const* d_out_all, void* const* d_ws_all, size_t ws_bytes) {
  if (nranks != NR || (unsigned)rank >= (unsigned)NR || ws_bytes < WS_NEED) return;
  Params p;
  p.x = (const unsigned char*)d_in[0];
  p.w = (const unsigned char*)d_in[1];
  p.sx = (const float*)d_in[2];
  p.sw = (const float*)d_in[3];
  p.out = (float*)d_out;
  for (int i = 0; i < NR; ++i) p.ws[i] = (unsigned char*)d_ws_all[i];
  p.myws = p.ws[rank];
  p.rank = rank;
  p.thr = envi("THR", 1);
  p.na = envi("NA", 20);
  p.cs = envi("CS", 0);
  p.ps = envi("PS", 1); p.te = envi("TE", 0); p.tl = envi("TL", 0); p.xa = 0;
  build_table(p);
  gemm_a2a<<<NWG, 512, 0, stream>>>(p);
  HIP_CHECK(hipGetLastError());
}
```

```cpp
#include <hip/hip_runtime.h>
#include <rccl/rccl.h>
#include <cstdio>

#define HIP_CHECK(expr) do { hipError_t _e = (expr); if (_e != hipSuccess) { \
    fprintf(stderr, "HIP error %s at %s:%d\n", hipGetErrorString(_e), __FILE__, __LINE__); \
    return; } } while (0)

namespace {

typedef int v2i __attribute__((ext_vector_type(2)));
typedef int v4i __attribute__((ext_vector_type(4)));
typedef unsigned v4u __attribute__((ext_vector_type(4)));
typedef int v8i __attribute__((ext_vector_type(8)));
typedef float v4f __attribute__((ext_vector_type(4)));
typedef unsigned long long u64;
#define LDS __attribute__((address_space(3)))
#define CST __attribute__((address_space(4)))
typedef __attribute__((address_space(1))) unsigned gu32;

constexpr int NR = 8;
constexpr int MLOC = 512, NN = 4096, KK = 4096, NSH = 512;
constexpr int STAGES = 5;
constexpr int XB = 8192, SB = 24576;
constexpr int NS = KK / 128;
constexpr int NA = 20;
constexpr int NWG = 256;
constexpr int LEAD = 4;
constexpr unsigned LOCAL_DELAY = 300;
constexpr u64 DEADLINE_TICKS = 100000000ull;

constexpr int HDR_B = 136;
constexpr int SLOT = 4352;
constexpr int UNITS = 256;
constexpr int PAIR_BYTES = UNITS * SLOT;
constexpr int MSG_OFF = SB * STAGES;
constexpr int MISC_OFF = MSG_OFF + 8 * SLOT;
constexpr int LDS_BYTES = MISC_OFF + 1024;
static_assert(LDS_BYTES <= 163840, "LDS");
constexpr unsigned WS_CTR = 0;
constexpr unsigned WS_TMO = 4096;
constexpr unsigned WS_XCC = 8192;
constexpr unsigned WS_STAGE = 1u << 20;
constexpr unsigned WS_XCH = WS_STAGE + 2u * NR * PAIR_BYTES;
constexpr unsigned WS_NEED = WS_XCH + 112u * 32768u;
constexpr int AUX_SYS = 17;
constexpr int AUX_WT = 16;

struct Params {
  const unsigned char* x;
  const unsigned char* w;
  unsigned char* myws;
  float* out;
  const float* sx;
  const float* sw;
  int rank, pad;
  unsigned char* ws[NR];
  unsigned pad2[2];
  unsigned tab[NWG];
};

template <int N_> __device__ __forceinline__ void wait_vm() { asm volatile("s_waitcnt vmcnt(%0)" :: "i"(N_) : "memory"); }
__device__ __forceinline__ void dma16(__amdgpu_buffer_rsrc_t rs, unsigned lds_dst, int voff, int soff) {
  asm volatile("s_mov_b32 m0, %0\n\ts_nop 0\n\tbuffer_load_dwordx4 %1, %2, %3 offen lds"
               :: "s"(lds_dst), "v"(voff), "s"(rs), "s"(soff) : "memory");
}
__device__ __forceinline__ void dma16_sys(__amdgpu_buffer_rsrc_t rs, unsigned lds_dst, int voff, int soff) {
  asm volatile("s_mov_b32 m0, %0\n\ts_nop 0\n\tbuffer_load_dwordx4 %1, %2, %3 offen sc0 sc1 lds"
               :: "s"(lds_dst), "v"(voff), "s"(rs), "s"(soff) : "memory");
}
#define SCHED() __builtin_amdgcn_sched_barrier(0)
#define BAR() do { SCHED(); asm volatile("s_barrier" ::: "memory"); SCHED(); } while (0)
#define LBAR() do { SCHED(); __builtin_amdgcn_s_waitcnt(0xC07F); asm volatile("s_barrier" ::: "memory"); SCHED(); } while (0)
__device__ __forceinline__ unsigned now32() { return (unsigned)__builtin_amdgcn_s_memrealtime(); }
__device__ __forceinline__ unsigned wave_scan(unsigned x) {
  x += __builtin_amdgcn_update_dpp(0u, x, 0x111, 0xf, 0xf, true);
  x += __builtin_amdgcn_update_dpp(0u, x, 0x112, 0xf, 0xf, true);
  x += __builtin_amdgcn_update_dpp(0u, x, 0x114, 0xf, 0xf, true);
  x += __builtin_amdgcn_update_dpp(0u, x, 0x118, 0xf, 0xf, true);
  x += __builtin_amdgcn_update_dpp(0u, x, 0x142, 0xa, 0xf, false);
  x += __builtin_amdgcn_update_dpp(0u, x, 0x143, 0xc, 0xf, false);
  return x;
}
__device__ __forceinline__ unsigned nonzero(unsigned v) { unsigned r; asm("v_min_u32 %0, 1, %1" : "=v"(r) : "v"(v)); return r; }
__device__ __forceinline__ bool expired(gu32* tmo, unsigned call, u64 ts) {
  if (__hip_atomic_load(tmo, __ATOMIC_RELAXED, __HIP_MEMORY_SCOPE_AGENT) == call ||
      __builtin_amdgcn_s_memrealtime() - ts > DEADLINE_TICKS) {
    __hip_atomic_store(tmo, call, __ATOMIC_RELAXED, __HIP_MEMORY_SCOPE_AGENT);
    return true;
  }
  return false;
}

struct Frags { v8i xf[2]; v8i wf[4]; };

__global__ void __launch_bounds__(512, 2) gemm_a2a(Params p) {
  __shared__ __attribute__((aligned(1024))) unsigned char smem[LDS_BYTES];
  const int tid = threadIdx.x, lane = tid & 63;
  const int wv8 = __builtin_amdgcn_readfirstlane(tid >> 6);
  const int wv = wv8 & 3;
  const int b = blockIdx.x;
  const unsigned smem_base = (unsigned)(size_t)(LDS unsigned char*)smem;
  unsigned td = p.tab[b];
  const unsigned char* px = p.x; const unsigned char* pw = p.w;
  unsigned char* myws = p.myws;
  asm volatile("" : "+s"(px), "+s"(pw), "+s"(td), "+s"(myws));
  const unsigned kind = td >> 30;
  const int n0 = kind == 2 ? NS - NA : 0;
  const int nsteps = n0 + (kind == 1 ? NA : NS);
  const int eidx = (td >> 18) & 127;
  const int r = lane & 15, g = lane >> 4;
  gu32* tmo = (gu32*)(size_t)(myws + WS_TMO);
  unsigned call, rbuf;

  if (wv8 >= 4) {
    __amdgpu_buffer_rsrc_t rx = __builtin_amdgcn_make_buffer_rsrc((void*)px, 0, MLOC * KK, 0x00020000);
    __amdgpu_buffer_rsrc_t rw = __builtin_amdgcn_make_buffer_rsrc((void*)pw, 0, KK * NN, 0x00020000);
    const int l3 = lane >> 3, l7 = lane & 7;
    const int c0 = (l7 ^ (l3 >> 1)) << 4;
    const int xo0 = (wv * 16 + l3) * KK + c0;
    const int xo1 = (xo0 + 8 * KK) ^ 64;
    const int wo0 = (wv * 32 + l3) * NN + c0 + (td & 31) * 128, wo1 = wo0 + 8 * NN;
    const int wo2 = (wo0 + 16 * NN) ^ 64, wo3 = wo2 + 8 * NN;
    int klo = kind == 2 ? NA : 0;
    const int khi = kind == 1 ? NA : NS;
    int kc = (td >> 13) & 31, left = khi - klo;
    int xrow = (((td >> 5) & 7) - (kind == 2 ? 4 : 0)) * 64 * KK;
    unsigned dst = smem_base + wv * 2048;
    auto issue = [&]() {
      const int so = kc * 128 * NN, sx = xrow + kc * 128;
      const unsigned dw = dst + XB + wv * 2048;
      dma16(rw, dw, wo0, so); dma16(rw, dw + 1024, wo1, so); dma16(rw, dw + 2048, wo2, so); dma16(rw, dw + 3072, wo3, so);
      dma16(rx, dst, xo0, sx); dma16(rx, dst + 1024, xo1, sx);
      if (++kc == khi) kc = klo;
      if (--left == 0) { klo = 0; kc = (td >> 8) & 31; xrow += 4 * 64 * KK; }
      dst += SB; if (dst >= smem_base + STAGES * SB) dst -= STAGES * SB;
    };
    if (kind == 0) {
      const unsigned tstart = now32();
      while (now32() - tstart < LOCAL_DELAY) __builtin_amdgcn_s_sleep(2);
    }
#pragma nounroll
    for (int kt = -STAGES; kt < nsteps - STAGES; ++kt) {
      if (kt >= -1) {
        wait_vm<6 * (STAGES - 2)>();
        BAR();
      }
      issue();
    }
    static_assert(STAGES == 5, "tail");
    wait_vm<18>(); BAR();
    wait_vm<12>(); BAR();
    wait_vm<6>(); BAR();
    wait_vm<0>();
    BAR();
    call = *(volatile LDS unsigned*)(smem + MISC_OFF);
    rbuf = dst + wv * (5120 - 2048);
  } else {
    const int dest = (td >> 2) & 7, tm = (td >> 5) & 7, tn = td & 3;
    unsigned ctr = *(const CST unsigned*)(size_t)(myws + WS_CTR + b * 4);
    float scx = *(const CST float*)(size_t)p.sx, scw = *(const CST float*)(size_t)p.sw;
    unsigned char* dptr = p.ws[dest];
    int rank = p.rank;
    asm volatile("" : "+s"(ctr), "+s"(scx), "+s"(scw), "+s"(dptr), "+s"(rank));
    float scale = scx * scw;
    call = ctr + 1;
    if (tid == 0) *(volatile LDS unsigned*)(smem + MISC_OFF) = call;
    const unsigned tag = call & 1;
    const unsigned here = call * 16 + (__builtin_amdgcn_s_getreg(6164) & 15);
    gu32* xcc_word = (gu32*)(size_t)(myws + WS_XCC) + eidx;
    if (kind == 1 && tid == 0) __hip_atomic_store(xcc_word, here, __ATOMIC_RELAXED, __HIP_MEMORY_SCOPE_AGENT);
    unsigned there = 0;
    const int wm = wv & 1, wn = wv >> 1;
    rbuf = smem_base + MSG_OFF + wv * 2 * SLOT;
    const int ucol0 = 4 * tn + 2 * wn, urow = tm * 2 + wm;
    dptr += WS_STAGE + (tag * NR + rank) * PAIR_BYTES + (urow * 16 + ucol0) * SLOT;
    int wa[4];
    {
      const int q = r >> 1, pp = r & 1, fl = ((q >> 1) & 3) | ((g & 1) << 2);
#pragma unroll
      for (int ni = 0; ni < 4; ++ni) wa[ni] = XB + (16 * g + q) * 128 + (((wn * 4 + ni) ^ fl) << 4) + pp * 8;
    }
    const int xa0 = (wm * 32 + r) * 128 + ((g ^ (r >> 1)) << 4);
    const int xa1 = xa0 ^ 64;
    auto rd = [&](Frags& f, int sb) {
#pragma unroll
      for (int mi = 0; mi < 2; ++mi) {
        v4i a = *(LDS v4i*)(smem + sb + xa0 + mi * 2048);
        v4i c = *(LDS v4i*)(smem + sb + xa1 + mi * 2048);
        f.xf[mi] = __builtin_shufflevector(a, c, 0, 1, 2, 3, 4, 5, 6, 7);
      }
#pragma unroll
      for (int ni = 0; ni < 4; ++ni) {
#pragma unroll
        for (int j = 0; j < 4; ++j) {
          v2i t = __builtin_amdgcn_ds_read_tr8_b64_v2i32((LDS v2i*)(smem + sb + wa[ni] + (j & 1) * 1024 + (j >> 1) * 8192));
          f.wf[ni][2 * j] = t[0]; f.wf[ni][2 * j + 1] = t[1];
        }
      }
    };
    v4f acc[8];
#pragma unroll
    for (int i = 0; i < 8; ++i) acc[i] = (v4f){0.f, 0.f, 0.f, 0.f};
    auto mma = [&](const Frags& f) {
#pragma unroll
      for (int ni = 0; ni < 4; ++ni)
#pragma unroll
        for (int mi = 0; mi < 2; ++mi) {
          const int i = (ni >> 1) * 4 + mi * 2 + (ni & 1);
          acc[i] = __builtin_amdgcn_mfma_scale_f32_16x16x128_f8f6f4(f.wf[ni], f.xf[mi], acc[i],  1,  0, 0, 0, 0, 0);
        }
    };
    auto interleave = [&]() {
#pragma unroll
      for (int i = 0; i < 8; ++i) {
        __builtin_amdgcn_sched_group_barrier(0x008, 1, 0);
        __builtin_amdgcn_sched_group_barrier(0x002, 2, 0);
        __builtin_amdgcn_sched_group_barrier(0x100, 4, 0);
      }
    };
    __amdgpu_buffer_rsrc_t rxch = __builtin_amdgcn_make_buffer_rsrc((void*)(myws + WS_XCH + eidx * 32768u + wv * 8192), 0, 8192, 0x00020000);
    v4u part[8];
#pragma unroll
    for (int i = 0; i < 8; ++i) part[i] = (v4u){0u, 0u, 0u, 0u};
    Frags f0, f1;
    LBAR();
    rd(f0, 0);
    int sb = SB;
#pragma nounroll
    for (int kt = 0; kt + 2 < nsteps; kt += 2) {
      if (kt + 4 == n0) there = __hip_atomic_load(xcc_word, __ATOMIC_RELAXED, __HIP_MEMORY_SCOPE_AGENT);
      if (kind == 1 && kt + LEAD == NA) {
#pragma unroll
        for (int i = 0; i < 8; ++i) part[i] = __builtin_amdgcn_raw_buffer_load_b128(rxch, i * 1024 + lane * 16, 0, AUX_WT);
      }
      LBAR();
      rd(f1, sb); mma(f0); interleave();
      sb += SB; if (sb == STAGES * SB) sb = 0;
      LBAR();
      rd(f0, sb); mma(f1); interleave();
      sb += SB; if (sb == STAGES * SB) sb = 0;
      if (kt + 2 == n0) {
        SCHED();
#pragma unroll
        for (int i = 0; i < 8; ++i) {
          v4u t = __builtin_bit_cast(v4u, acc[i] + 0.0f);
          asm("" : "+v"(t));
          acc[i] = __builtin_bit_cast(v4f, t ^ 0x80000000u);
        }
        if (__builtin_amdgcn_readfirstlane(there) == here) {
#pragma unroll
          for (int i = 0; i < 8; ++i) __builtin_amdgcn_raw_buffer_store_b128(__builtin_bit_cast(v4u, acc[i]), rxch, i * 1024 + lane * 16, 0, 0);
        } else {
#pragma unroll
          for (int i = 0; i < 8; ++i) __builtin_amdgcn_raw_buffer_store_b128(__builtin_bit_cast(v4u, acc[i]), rxch, i * 1024 + lane * 16, 0, AUX_WT);
        }
#pragma unroll
        for (int i = 0; i < 8; ++i) acc[i] = (v4f){0.f, 0.f, 0.f, 0.f};
      }
    }
    LBAR();
    rd(f1, sb); mma(f0); interleave();
    SCHED();
    mma(f1);
    SCHED();
    asm volatile("" :: "v"(there));
    if (kind == 1) {
      auto stale = [&]() {
        unsigned x = ~0u;
#pragma unroll
        for (int i = 0; i < 8; ++i) x = min(min(x, min(part[i][0], part[i][1])), min(part[i][2], part[i][3]));
        return __builtin_amdgcn_ballot_w64(x == 0) != 0;
      };
      if (__builtin_expect(stale(), 0)) {
        const u64 ts = __builtin_amdgcn_s_memrealtime();
        bool dead = false;
        do {
          for (unsigned spin = 1; !dead && __builtin_amdgcn_readfirstlane(__builtin_amdgcn_raw_buffer_load_b32(rxch, 8188, 0, AUX_WT)) == 0; ++spin) {
            if ((spin & 15) == 0) dead = expired(tmo, call, ts);
            __builtin_amdgcn_s_sleep(2);
          }
#pragma unroll
          for (int i = 0; i < 8; ++i) part[i] = __builtin_amdgcn_raw_buffer_load_b128(rxch, i * 1024 + lane * 16, 0, AUX_WT);
          if (!stale()) break;
          dead = dead || expired(tmo, call, ts);
        } while (!dead);
        if (dead) scale = __builtin_nanf("");
      }
#pragma unroll
      for (int i = 0; i < 8; ++i) acc[i] -= __builtin_bit_cast(v4f, part[i]);
    }
    if (kind == 0) {
      __amdgpu_buffer_rsrc_t ro = __builtin_amdgcn_make_buffer_rsrc((void*)p.out, 0, NR * MLOC * NSH * 4, 0x00020000);
      const int obase = ((rank * MLOC + urow * 32 + r) * NSH + ucol0 * 32 + g * 4) * 4;
#pragma unroll
      for (int i = 0; i < 8; ++i) {
        v4f y = acc[i] * scale;
#pragma unroll
        for (int j = 0; j < 4; ++j) y[j] = (y[j] <= 0.f) ? 0.f : y[j];
        __builtin_amdgcn_raw_buffer_store_b128(__builtin_bit_cast(v4u, y), ro, obase + ((i >> 1) & 1) * 16 * NSH * 4 + (i >> 2) * 128 + (i & 1) * 64, 0, AUX_WT);
      }
    } else {
      __amdgpu_buffer_rsrc_t rd_ = __builtin_amdgcn_make_buffer_rsrc((void*)dptr, 0, 2 * SLOT, 0x00020000);
      auto send_chunk = [&](int h, int sent, int total) {
        const int c = sent + lane;
        v4u v = *(LDS v4u*)(smem + MSG_OFF + (wv * 2 + h) * SLOT + c * 16);
        if (total - sent >= 24) wait_vm<0>();
        __builtin_amdgcn_raw_buffer_store_b128(v, rd_, c < total ? h * SLOT + c * 16 : 2 * SLOT, 0, AUX_SYS);
      };
      auto pack = [&](const v4f* a4, unsigned msg) -> int {
        float y[16]; u64 m[16];
        unsigned hm = 0;
#pragma unroll
        for (int i = 0; i < 4; ++i) {
          const v4f t = a4[i] * scale;
#pragma unroll
          for (int j = 0; j < 4; ++j) {
            y[i * 4 + j] = t[j];
            m[i * 4 + j] = __builtin_amdgcn_ballot_w64(!(t[j] <= 0.f));
          }
        }
#define AC(n) "v_addc_co_u32 %0, vcc, %0, %0, %" #n "\n\t"
        asm("s_nop 1\n\t" AC(1) AC(2) AC(3) AC(4) AC(5) AC(6) AC(7) AC(8) AC(9) AC(10) AC(11) AC(12) AC(13) AC(14) AC(15) AC(16) "s_nop 1"
            : "+v"(hm) : "s"(m[0]), "s"(m[1]), "s"(m[2]), "s"(m[3]), "s"(m[4]), "s"(m[5]), "s"(m[6]), "s"(m[7]),
              "s"(m[8]), "s"(m[9]), "s"(m[10]), "s"(m[11]), "s"(m[12]), "s"(m[13]), "s"(m[14]), "s"(m[15]) : "vcc");
#undef AC
        const unsigned cnt = __builtin_popcount(hm);
        const unsigned incl = wave_scan(cnt);
        const unsigned nval = __builtin_amdgcn_readlane(incl, 63);
        unsigned ad = msg + (incl - cnt) * 4;
#define WR(mm, yy) "s_mov_b64 exec, %" #mm "\n\tds_write_b32 %0, %" #yy " offset:136\n\tv_add_u32 %0, 4, %0\n\t"
#pragma unroll
        for (int i = 0; i < 16; i += 8)
          asm volatile(WR(1, 9) WR(2, 10) WR(3, 11) WR(4, 12) WR(5, 13) WR(6, 14) WR(7, 15) WR(8, 16) "s_mov_b64 exec, -1"
                       : "+v"(ad) : "s"(m[i]), "s"(m[i + 1]), "s"(m[i + 2]), "s"(m[i + 3]), "s"(m[i + 4]), "s"(m[i + 5]), "s"(m[i + 6]), "s"(m[i + 7]),
                         "v"(y[i]), "v"(y[i + 1]), "v"(y[i + 2]), "v"(y[i + 3]), "v"(y[i + 4]), "v"(y[i + 5]), "v"(y[i + 6]), "v"(y[i + 7]) : "memory");
#undef WR
        static_assert(HDR_B == 136, "offset above");
        *(LDS unsigned short*)(size_t)(msg + (lane & 31) * 4 + (lane >> 5) * 2) = (unsigned short)hm;
        const u64 nz = __builtin_amdgcn_ballot_w64(hm != 0);
        const unsigned z = (unsigned)nz | (unsigned)(nz >> 32);
        asm volatile("s_mov_b64 exec, 31\n\tds_write_b32 %0, %1 offset:128\n\ts_mov_b64 exec, -1"
                     :: "v"(msg + lane * 4 + (lane < 2 ? 0 : nval * 4)), "v"(lane < 2 ? (((z >> (lane * 16)) & 0xffffu) | 0x10000u) : 0u) : "memory");
        return (HDR_B + nval * 4 + 15) >> 4;
      };
      const int tot0 = pack(acc, rbuf);
      send_chunk(0, 0, tot0);
      const int tot1 = pack(acc + 4, rbuf + SLOT);
#pragma nounroll
      for (int c = 64; c < tot0; c += 64) send_chunk(0, c, tot0);
#pragma nounroll
      for (int c = 0; c < tot1; c += 64) send_chunk(1, c, tot1);
      if (kind == 1) {
#pragma unroll
        for (int i = 0; i < 8; ++i) __builtin_amdgcn_raw_buffer_store_b128((v4u){0u, 0u, 0u, 0u}, rxch, i * 1024 + lane * 16, 0, AUX_WT);
      }
    }
    if (tid == 0)
      __hip_atomic_store((gu32*)(size_t)(myws + WS_CTR) + b, call, __ATOMIC_RELAXED, __HIP_MEMORY_SCOPE_AGENT);
  }
  call = __builtin_amdgcn_readfirstlane(call);
  rbuf = __builtin_amdgcn_readfirstlane(rbuf);
#pragma nounroll
  for (int late = wv8 >= 4 ? 0 : 1; late < (kind == 1 ? 2 : (kind == 2 ? 1 : 0)); ++late) {
    const int idx = late ? eidx * 8 + wv8 : (eidx + (kind - 1) * 112) * 4 + wv;
    const int q = idx / 7, src = (p.rank + 1 + idx - q * 7) & 7;
    const int un = q + late * 128;
    __amdgpu_buffer_rsrc_t rs = __builtin_amdgcn_make_buffer_rsrc(
        (void*)(myws + WS_STAGE + ((call & 1) * NR + src) * PAIR_BYTES + un * SLOT), 0, SLOT, 0x00020000);
    __amdgpu_buffer_rsrc_t ro = __builtin_amdgcn_make_buffer_rsrc((void*)p.out, 0, NR * MLOC * NSH * 4, 0x00020000);
    const u64 ts = __builtin_amdgcn_s_memrealtime();
    bool dead = false;
    unsigned hdr;
    const int hoff = lane < 34 ? lane * 4 : SLOT;
    for (unsigned spin = 1;; ++spin) {
      hdr = __builtin_amdgcn_raw_buffer_load_b32(rs, hoff, 0, AUX_SYS);
      const u64 nz = __builtin_amdgcn_ballot_w64(hdr != 0);
      const unsigned z = ((unsigned)__builtin_amdgcn_readlane(hdr, 32) & 0xffffu) | ((unsigned)__builtin_amdgcn_readlane(hdr, 33) << 16);
      if ((unsigned)(nz >> 32) == 3u && (unsigned)nz == z) break;
      if ((spin & 255) == 0 && expired(tmo, call, ts)) { dead = true; hdr = 0; break; }
      __builtin_amdgcn_s_sleep(4);
    }
    dma16_sys(rs, rbuf, lane * 16, 0); dma16_sys(rs, rbuf + 1024, lane * 16, 1024); dma16_sys(rs, rbuf + 2048, lane * 16, 2048);
    const unsigned lo = __builtin_amdgcn_permlane32_swap(hdr, hdr, false, false)[0];
    const unsigned hm = lane < 32 ? (lo & 0xffffu) : (lo >> 16);
    const unsigned cnt = __builtin_popcount(hm);
    const unsigned incl = wave_scan(cnt);
    const unsigned endb = HDR_B + __builtin_amdgcn_readlane(incl, 63) * 4;
    const unsigned first = HDR_B + (incl - cnt) * 4;
    for (unsigned c = 3072; c <= endb; c += 1024) dma16_sys(rs, rbuf + c, lane * 16, c);
    unsigned ad[16];
    {
      unsigned cur = rbuf + first;
#pragma unroll
      for (int i = 0; i < 16; ++i) {
        const bool keep = (hm >> (15 - i)) & 1;
        ad[i] = keep ? cur : rbuf + endb;
        cur += keep ? 4 : 0;
      }
    }
    unsigned v[16];
    for (unsigned spin = 1;; ++spin) {
      wait_vm<0>();
      unsigned got = 0;
#pragma unroll
      for (int i = 0; i < 16; ++i) v[i] = *(volatile LDS unsigned*)(size_t)ad[i];
#pragma unroll
      for (int i = 0; i < 16; ++i) got += nonzero(v[i]);
      const u64 miss = __builtin_amdgcn_ballot_w64(got != cnt);
      if (!miss) break;
      if ((spin & 63) == 0 && expired(tmo, call, ts)) { dead = true; break; }
      for (unsigned c = __builtin_amdgcn_readlane(first, __builtin_ctzll(miss)) & ~1023u; c < endb; c += 1024) dma16_sys(rs, rbuf + c, lane * 16, c);
    }
    if (dead) {
#pragma unroll
      for (int i = 0; i < 16; ++i) v[i] = 0x7fc00000u;
    }
    const int obase = ((src * MLOC + (un >> 4) * 32 + r) * NSH + (un & 15) * 32 + g * 4) * 4;
#pragma unroll
    for (int i = 0; i < 4; ++i) {
      v4u y = {v[i * 4], v[i * 4 + 1], v[i * 4 + 2], v[i * 4 + 3]};
      __builtin_amdgcn_raw_buffer_store_b128(y, ro, obase + (i >> 1) * 16 * NSH * 4 + (i & 1) * 64, 0, AUX_WT);
    }
    for (unsigned c = lane * 16; c < endb; c += 1024)
      __builtin_amdgcn_raw_buffer_store_b128((v4u){0u, 0u, 0u, 0u}, rs, c, 0, AUX_SYS);
  }
}

void build_table(Params& p) {
  int eidx = 0;
  for (int g = 0; g < 8; ++g) {
    const int h = g >> 1, ne = (g & 1) ? 12 : 16;
    for (int i = 0; i < 32; ++i) {
      unsigned t;
      if (i >= 2 * ne) {
        const int tm = i - 2 * ne, rot = ((tm + g) & 7) * 4;
        t = (unsigned)(p.rank * 4 + h) | (tm << 5) | (rot << 8) | (rot << 13);
      } else {
        const int e = i % ne, d = (3 * h + ((g & 1) ? 0 : 3) + (e >> 2)) % 7, tm = e & 3, r4 = (tm + g) & 3;
        t = (unsigned)(((p.rank + 1 + d) & 7) * 4 + ((h + d) & 3)) | ((eidx + e) << 18);
        if (i < ne) t |= (tm << 5) | ((r4 * NA / 4) << 13) | (1u << 30);
        else t |= ((tm + 4) << 5) | ((g * 4) << 8) | ((NA + r4 * (NS - NA) / 4) << 13) | (2u << 30);
      }
      p.tab[i * 8 + g] = t;
    }
    eidx += ne;
  }
}

}

extern "C" void kernel_launch(int rank, int nranks, ncclComm_t comm, hipStream_t stream,
                              void* const* d_in, const int* in_sizes, int n_in,
                              void* d_out, int out_size, const int* in_dtypes, int out_dtype,
                              void* const* d_out_all, void* const* d_ws_all, size_t ws_bytes) {
  if (nranks != NR || (unsigned)rank >= (unsigned)NR || ws_bytes < WS_NEED) return;
  Params p;
  p.x = (const unsigned char*)d_in[0];
  p.w = (const unsigned char*)d_in[1];
  p.sx = (const float*)d_in[2];
  p.sw = (const float*)d_in[3];
  p.out = (float*)d_out;
  for (int i = 0; i < NR; ++i) p.ws[i] = (unsigned char*)d_ws_all[i];
  p.myws = p.ws[rank];
  p.rank = rank;
  p.pad = 0; p.pad2[0] = p.pad2[1] = 0;
  build_table(p);
  gemm_a2a<<<NWG, 512, 0, stream>>>(p);
  HIP_CHECK(hipGetLastError());
}
```
